```python
import numpy as np

kernel_name = "gpu_gemm_rs_m4096_k4096_n16384_e4m3e4m3_dequant"


ABSMAX_RTOL = 0.03

COMPUTE_DTYPE = "float32"
EPILOGUE = "dequant"
IN_QUANT = None
OUT_QUANT = None
IN_FP8 = "e4m3e4m3"
IN_DTYPES = {"x": "e4m3", "w_mat": "e4m3",
             "scale_x": "float32", "scale_w": "float32"}
WORLD_SIZE = 8

def _snap_fp8(x, mant_bits, max_abs, min_normal):
    a = np.abs(x.astype(np.float64))
    _m, _e = np.frexp(a)
    q = np.where(a >= min_normal, np.ldexp(1.0, _e - 1 - mant_bits),
                 min_normal * 2.0 ** -mant_bits)
    snapped = np.minimum(np.round(a / q) * q, max_abs)
    return (np.sign(x) * snapped).astype(np.float32)

def setup_inputs():

    x = _snap_fp8(np.random.randn(4096, 4096).astype(np.float32), 3, 448.0, 2.0 ** -6)
    w_mat = _snap_fp8(np.random.randn(4096, 16384).astype(np.float32), 3, 448.0, 2.0 ** -6)
    scale_x = (0.02 * (1.0 + np.random.rand(1))).astype(np.float32)
    scale_w = (0.02 * (1.0 + np.random.rand(1))).astype(np.float32)
    return {"x": x, "w_mat": w_mat, "scale_x": scale_x, "scale_w": scale_w}

def reference(x, w_mat, scale_x, scale_w):
    acc = (x @ w_mat).astype(np.float32)
    y = (acc * (scale_x * scale_w)).astype(np.float32)
    return y
```

```cpp
#include <hip/hip_runtime.h>
#include <rccl/rccl.h>
#include <cstdio>
#include <cstdint>

#define HIP_CHECK(expr) do { hipError_t _e = (expr); if (_e != hipSuccess) { \
    fprintf(stderr, "HIP error %s at %s:%d\n", hipGetErrorString(_e), __FILE__, __LINE__); \
    return; } } while (0)

namespace {

typedef int v8i __attribute__((ext_vector_type(8)));
typedef float v4f __attribute__((ext_vector_type(4)));
typedef unsigned v4u __attribute__((ext_vector_type(4)));

constexpr int NR = 8;
constexpr int GM = 4096, GK = 4096, GN = 16384;
constexpr int KS = GK / NR;
constexpr int NS = GN / NR;
constexpr int MS = GM / NR;
constexpr int TM = 128, TN = 256, BK = 128;
constexpr int NMB = GM / TM;
constexpr int NNB = NS / TN;
constexpr int NTILES = NMB * NNB;
constexpr int NKT = GK / BK;

constexpr int GRID = 256, THREADS = 512;
#define LINK_WGS 3
constexpr int LW = LINK_WGS;
constexpr int NCOMM = NR * LW;
constexpr int NPOOL = GRID - NCOMM + LW;

constexpr size_t XS_BYTES = (size_t)GM * GK;
constexpr size_t WT_BYTES = (size_t)NS * GK;
constexpr size_t STAGE_BYTES = XS_BYTES + WT_BYTES;
constexpr size_t OFF_Y = 2 * STAGE_BYTES;
constexpr size_t Y_BYTES = (size_t)NTILES * TM * TN * 4;
constexpr size_t OFF_CTL = OFF_Y + Y_BYTES;
constexpr size_t WS_NEED = OFF_CTL + 65536;
constexpr int CW = 32;
constexpr int C_CALL = 0 * CW, C_TILE = 1 * CW, C_TMO = 2 * CW, C_YCNT = 3 * CW;
constexpr int C_XCNT = 8 * CW;
constexpr int C_WCNT = 40 * CW;
constexpr int C_YFLAG = 64 * CW;

constexpr int XU = 4;
constexpr int U_W0 = 8, U_X0 = U_W0 + 8 * XU, U_W1 = U_X0 + 56, U_XW = U_W1 + 24 * XU;
constexpr int YU = 8;
constexpr int U_ALL = U_XW + 32 * YU;
constexpr unsigned XCNT_PER_CALL = NR * XU;
constexpr unsigned WCNT_PER_CALL = NR * 8;
constexpr unsigned YCNT_PER_CALL = (NR - 1) * 32 * YU;

constexpr int STAGE_LDS = (TN + TM) * BK;
constexpr int LDS_CTL = 3 * STAGE_LDS;
constexpr int LDS_BYTES = LDS_CTL + 64;

constexpr unsigned long long DEADLINE_TICKS = 50000000ull;

struct Params {
    const uint8_t* x;
    const uint8_t* w;
    const float* sx;
    const float* sw;
    float* out[NR];
    uint8_t* ws[NR];
    int rank;
};

template <bool SYS>
__device__ __forceinline__ void wait_ge(unsigned* p, unsigned target, unsigned* tmo, unsigned call) {
    unsigned long long t0 = __builtin_amdgcn_s_memrealtime();
    for (;;) {
        unsigned v = SYS ? __hip_atomic_load(p, __ATOMIC_RELAXED, __HIP_MEMORY_SCOPE_SYSTEM)
                         : __hip_atomic_load(p, __ATOMIC_RELAXED, __HIP_MEMORY_SCOPE_AGENT);
        if (v >= target) break;
        if (__hip_atomic_load(tmo, __ATOMIC_RELAXED, __HIP_MEMORY_SCOPE_AGENT) == call ||
            __builtin_amdgcn_s_memrealtime() - t0 > DEADLINE_TICKS) {
            __hip_atomic_store(tmo, call, __ATOMIC_RELAXED, __HIP_MEMORY_SCOPE_AGENT);
            break;
        }
        __builtin_amdgcn_s_sleep(2);
    }
}

__device__ __forceinline__ void tr4x4(unsigned r0, unsigned r1, unsigned r2, unsigned r3,
                                      unsigned& o0, unsigned& o1, unsigned& o2, unsigned& o3) {
    unsigned t0 = __builtin_amdgcn_perm(r1, r0, 0x05010400u);
    unsigned t1 = __builtin_amdgcn_perm(r1, r0, 0x07030602u);
    unsigned t2 = __builtin_amdgcn_perm(r3, r2, 0x05010400u);
    unsigned t3 = __builtin_amdgcn_perm(r3, r2, 0x07030602u);
    o0 = __builtin_amdgcn_perm(t2, t0, 0x05040100u);
    o1 = __builtin_amdgcn_perm(t2, t0, 0x07060302u);
    o2 = __builtin_amdgcn_perm(t3, t1, 0x05040100u);
    o3 = __builtin_amdgcn_perm(t3, t1, 0x07060302u);
}

#define VMCNT0() asm volatile("s_waitcnt vmcnt(0)" ::: "memory")

__device__ __forceinline__ void comm_role(const Params& P, unsigned call, int link, int wl, int lane) {
    const int rank = P.rank;
    const int p = link;
    uint8_t* myws = P.ws[rank];
    unsigned* myctl = (unsigned*)(myws + OFF_CTL);
    unsigned* tmo = myctl + C_TMO;
    uint8_t* dstage = P.ws[p] + (size_t)(call & 1u) * STAGE_BYTES;
    unsigned* dctl = (unsigned*)(P.ws[p] + OFF_CTL);
    auto rs_stage = __builtin_amdgcn_make_buffer_rsrc(dstage, 0, (int)STAGE_BYTES, 0x00020000);
    auto rs_out = __builtin_amdgcn_make_buffer_rsrc(P.out[p], 0, (int)((size_t)MS * GN * 4), 0x00020000);
    const int nunits = (p == rank) ? U_XW : U_ALL;
    for (int u = wl; u < nunits; u += 8 * LW) {
        unsigned* cnt;
        if (u < U_W0 || (u >= U_X0 && u < U_W1)) {
            const int wu = (u < U_W0) ? u : (u - U_X0 + 8);
            const int nb = wu >> 3, kb = (wu >> 1) & 3, nh = wu & 1;
            const int kg = lane & 7, ng = lane >> 3;
            const uint8_t* src = P.w + (size_t)(kb * 128 + kg * 16) * GN + (p * NS + nb * TN + nh * 128 + ng * 16);
            v4u in[16];
#pragma unroll
            for (int i = 0; i < 16; ++i) in[i] = *(const v4u*)(src + (size_t)i * GN);
            v4u o[16];
#pragma unroll
            for (int a = 0; a < 4; ++a)
#pragma unroll
                for (int b = 0; b < 4; ++b) {
                    unsigned q0, q1, q2, q3;
                    tr4x4(in[4 * a + 0][b], in[4 * a + 1][b], in[4 * a + 2][b], in[4 * a + 3][b], q0, q1, q2, q3);
                    o[4 * b + 0][a] = q0; o[4 * b + 1][a] = q1; o[4 * b + 2][a] = q2; o[4 * b + 3][a] = q3;
                }
            const int voff = (int)XS_BYTES + (nb * TN + nh * 128 + ng * 16) * GK + rank * KS + kb * 128 + kg * 16;
#pragma unroll
            for (int j = 0; j < 16; ++j)
                __builtin_amdgcn_raw_buffer_store_b128(o[j], rs_stage, voff + j * GK, 0, 17);
            cnt = dctl + C_WCNT + nb * CW;
        } else if (u < U_XW) {
            const int xu = (u < U_X0) ? (u - U_W0) : (u - U_W1 + 8 * XU);
            const int blk = xu / XU, sub = xu % XU;
            const int owner = (p + 1 + (blk & 7)) & 7;
            const int mb = owner * 4 + (blk >> 3);
            const int row0 = mb * TM + sub * 32 + (lane >> 5);
            const uint8_t* src = P.x + (size_t)row0 * KS + (lane & 31) * 16;
            v4u in[16];
#pragma unroll
            for (int i = 0; i < 16; ++i) in[i] = *(const v4u*)(src + (size_t)i * 2 * KS);
            const int voff = row0 * GK + rank * KS + (lane & 31) * 16;
#pragma unroll
            for (int i = 0; i < 16; ++i)
                __builtin_amdgcn_raw_buffer_store_b128(in[i], rs_stage, voff + i * 2 * GK, 0, 17);
            cnt = dctl + C_XCNT + mb * CW;
        } else {
            const int yu = u - U_XW;
            const int tl = yu / YU, sub = yu % YU;
            const int rnd = tl >> 3, nb = tl & 7;
            const int tile = (p * 4 + rnd) * NNB + nb;
            if (lane == 0) wait_ge<false>(myctl + C_YFLAG + tile * CW, call, tmo, call);
            __builtin_amdgcn_fence(__ATOMIC_ACQUIRE, "agent");
            const bool bad = __hip_atomic_load(tmo, __ATOMIC_RELAXED, __HIP_MEMORY_SCOPE_AGENT) == call;
            const uint8_t* src = myws + OFF_Y + (size_t)tile * (TM * TN * 4) + (size_t)sub * 16384 + lane * 16;
            v4u in[16];
#pragma unroll
            for (int i = 0; i < 16; ++i) in[i] = *(const v4u*)(src + i * 1024);
            if (bad) {
#pragma unroll
                for (int i = 0; i < 16; ++i) in[i] = (v4u){0x7fc00000u, 0x7fc00000u, 0x7fc00000u, 0x7fc00000u};
            }
            const int voff = ((rnd * TM + sub * 16) * GN + rank * NS + nb * TN) * 4 + lane * 16;
#pragma unroll
            for (int i = 0; i < 16; ++i)
                __builtin_amdgcn_raw_buffer_store_b128(in[i], rs_out, voff + i * (GN * 4), 0, 17);
            cnt = dctl + C_YCNT;
        }
        VMCNT0();
        if (lane == 0) __hip_atomic_fetch_add(cnt, 1u, __ATOMIC_RELAXED, __HIP_MEMORY_SCOPE_SYSTEM);
    }
}

__global__ __launch_bounds__(THREADS, 2) void fused_kernel(Params P) {
    extern __shared__ __attribute__((aligned(16))) uint8_t smem[];
    const int tid = threadIdx.x;
    const int wave = __builtin_amdgcn_readfirstlane(tid >> 6);
    const int lane = tid & 63;
    const int rank = P.rank;
    uint8_t* myws = P.ws[rank];
    unsigned* myctl = (unsigned*)(myws + OFF_CTL);
    unsigned* tmo = myctl + C_TMO;
    volatile unsigned* sctl = (volatile unsigned*)(smem + LDS_CTL);

    if (tid == 0) {
        unsigned t = __hip_atomic_fetch_add(myctl + C_CALL, 1u, __ATOMIC_RELAXED, __HIP_MEMORY_SCOPE_AGENT);
        sctl[0] = t / GRID + 1u;
    }
    __syncthreads();
    const unsigned call = __builtin_amdgcn_readfirstlane(sctl[0]);

    const int wg = blockIdx.x;
    bool gemm_worker = true;
    if (wg < NCOMM) {
        const int link = wg & 7, sub = wg >> 3;
        comm_role(P, call, link, sub * 8 + wave, lane);
        gemm_worker = (link == rank);
        if (!gemm_worker) {
            if (wg == ((rank + 1) & 7)) {
                __syncthreads();
                if (tid == 0) {
                    wait_ge<true>(myctl + C_YCNT, call * YCNT_PER_CALL, tmo, call);
                    __builtin_amdgcn_fence(__ATOMIC_ACQUIRE, "");
                    if (__hip_atomic_load(tmo, __ATOMIC_RELAXED, __HIP_MEMORY_SCOPE_AGENT) == call)
                        P.out[rank][0] = __builtin_nanf("");
                }
            }
            return;
        }
    }

    const float scale = P.sx[0] * P.sw[0];
    const uint8_t* stage = myws + (size_t)(call & 1u) * STAGE_BYTES;
    const int wn = wave >> 1, wm = wave & 1;
    const int fr = lane & 15, fq = lane >> 4;
    const int sw0 = ((fq) ^ (fr & 7)) * 16;
    const int a_off = (wn * 64 + fr) * BK + sw0;
    const int b_off = TN * BK + (wm * 64 + fr) * BK + sw0;
    const int srow = lane >> 3;
    const int schunk = ((lane & 7) ^ (srow & 7)) * 16;

    for (;;) {
        __syncthreads();
        if (tid == 0) {
            unsigned t = __hip_atomic_fetch_add(myctl + C_TILE, 1u, __ATOMIC_RELAXED, __HIP_MEMORY_SCOPE_AGENT);
            t -= (call - 1u) * (unsigned)(NTILES + NPOOL);
            sctl[1] = t;
            if (t < (unsigned)NTILES) {
                int rnd, oi, nb;
                if (t < 64) { rnd = 0; nb = t >> 3; oi = t & 7; }
                else { int j = t - 64; rnd = 1 + (j >> 6); oi = (j >> 3) & 7; nb = j & 7; }
                const int mb = ((rank + 1 + oi) & 7) * 4 + rnd;
                wait_ge<true>(myctl + C_WCNT + nb * CW, call * WCNT_PER_CALL, tmo, call);
                wait_ge<true>(myctl + C_XCNT + mb * CW, call * XCNT_PER_CALL, tmo, call);
                __builtin_amdgcn_fence(__ATOMIC_ACQUIRE, "");
                VMCNT0();
            }
        }
        __syncthreads();
        const unsigned t = __builtin_amdgcn_readfirstlane(sctl[1]);
        if (t >= (unsigned)NTILES) break;
        int rnd, oi, nb;
        if (t < 64) { rnd = 0; nb = t >> 3; oi = t & 7; }
        else { int j = t - 64; rnd = 1 + (j >> 6); oi = (j >> 3) & 7; nb = j & 7; }
        const int owner = (rank + 1 + oi) & 7;
        const int mb = owner * 4 + rnd;

        const uint8_t* gA = stage + XS_BYTES + (size_t)(nb * TN + wave * 32 + srow) * GK + schunk;
        const uint8_t* gB = stage + (size_t)(mb * TM + wave * 16 + srow) * GK + schunk;
        auto issue = [&](int kt, int buf) {
            uint8_t* la = smem + buf + wave * 4096;
            uint8_t* lb = smem + buf + TN * BK + wave * 2048;
            const uint8_t* a = gA + kt * BK;
            const uint8_t* b = gB + kt * BK;
#pragma unroll
            for (int j = 0; j < 4; ++j)
                __builtin_amdgcn_global_load_lds((const unsigned*)(a + (size_t)j * 8 * GK), (unsigned*)(la + j * 1024), 16, 0, 0);
#pragma unroll
            for (int j = 0; j < 2; ++j)
                __builtin_amdgcn_global_load_lds((const unsigned*)(b + (size_t)j * 8 * GK), (unsigned*)(lb + j * 1024), 16, 0, 0);
        };

        v4f acc[4][4];
#pragma unroll
        for (int i = 0; i < 4; ++i)
#pragma unroll
            for (int j = 0; j < 4; ++j) acc[i][j] = (v4f){0.f, 0.f, 0.f, 0.f};

        issue(0, 0);
        issue(1, STAGE_LDS);
        int cur = 0, nxt = 2 * STAGE_LDS;
        for (int kt = 0; kt < NKT; ++kt) {
            if (kt < NKT - 1) asm volatile("s_waitcnt vmcnt(6)" ::: "memory");
            else asm volatile("s_waitcnt vmcnt(0)" ::: "memory");
            __builtin_amdgcn_s_barrier();
            asm volatile("" ::: "memory");
            if (kt + 2 < NKT) issue(kt + 2, nxt);
            v8i af[4], bf[4];
#pragma unroll
            for (int i = 0; i < 4; ++i) {
                const v4u lo = *(const v4u*)(smem + cur + a_off + i * 2048);
                const v4u hi = *(const v4u*)(smem + cur + (a_off ^ 64) + i * 2048);
                af[i] = (v8i){(int)lo[0], (int)lo[1], (int)lo[2], (int)lo[3], (int)hi[0], (int)hi[1], (int)hi[2], (int)hi[3]};
            }
#pragma unroll
            for (int i = 0; i < 4; ++i) {
                const v4u lo = *(const v4u*)(smem + cur + b_off + i * 2048);
                const v4u hi = *(const v4u*)(smem + cur + (b_off ^ 64) + i * 2048);
                bf[i] = (v8i){(int)lo[0], (int)lo[1], (int)lo[2], (int)lo[3], (int)hi[0], (int)hi[1], (int)hi[2], (int)hi[3]};
            }
#pragma unroll
            for (int i = 0; i < 4; ++i)
#pragma unroll
                for (int j = 0; j < 4; ++j)
                    acc[i][j] = __builtin_amdgcn_mfma_scale_f32_16x16x128_f8f6f4(af[i], bf[j], acc[i][j], 0, 0, 0,
                                                                                 0x7F7F7F7F, 0, 0x7F7F7F7F);
            nxt = cur;
            cur += STAGE_LDS;
            if (cur == 3 * STAGE_LDS) cur = 0;
        }

        const bool bad = __hip_atomic_load(tmo, __ATOMIC_RELAXED, __HIP_MEMORY_SCOPE_AGENT) == call;
        const float s = bad ? __builtin_nanf("") : scale;
        if (owner == rank) {
            float* dst = P.out[rank] + (size_t)(rnd * TM + wm * 64 + fr) * GN + rank * NS + nb * TN + wn * 64 + fq * 4;
#pragma unroll
            for (int j = 0; j < 4; ++j)
#pragma unroll
                for (int i = 0; i < 4; ++i)
                    *(v4f*)(dst + (size_t)j * 16 * GN + i * 16) = acc[i][j] * s;
        } else {
            const int tile = mb * NNB + nb;
            auto rs_y = __builtin_amdgcn_make_buffer_rsrc(myws + OFF_Y + (size_t)tile * (TM * TN * 4), 0, TM * TN * 4, 0x00020000);
            const int voff = ((wm * 64 + fr) * TN + wn * 64 + fq * 4) * 4;
#pragma unroll
            for (int j = 0; j < 4; ++j)
#pragma unroll
                for (int i = 0; i < 4; ++i) {
                    v4f v = acc[i][j] * s;
                    __builtin_amdgcn_raw_buffer_store_b128(__builtin_bit_cast(v4u, v), rs_y, voff + (j * 16 * TN + i * 16) * 4, 0, 16);
                }
            VMCNT0();
            __syncthreads();
            if (tid == 0)
                __hip_atomic_store(myctl + C_YFLAG + tile * CW, call, __ATOMIC_RELAXED, __HIP_MEMORY_SCOPE_AGENT);
        }
    }
}

}

extern "C" void kernel_launch(int rank, int nranks, ncclComm_t comm,
                              hipStream_t stream,
                              void* const* d_in, const int* in_sizes, int n_in,
                              void* d_out, int out_size,
                              const int* in_dtypes, int out_dtype,
                              void* const* d_out_all,
                              void* const* d_ws_all, size_t ws_bytes) {
    (void)comm; (void)in_sizes; (void)n_in; (void)in_dtypes; (void)out_dtype; (void)d_out;
    if (nranks != NR || (unsigned)rank >= (unsigned)NR || ws_bytes < WS_NEED || out_size != MS * GN) {
        fprintf(stderr, "unsupported configuration\n");
        return;
    }
    static bool init = false;
    if (!init) {
        HIP_CHECK(hipFuncSetAttribute((const void*)fused_kernel, hipFuncAttributeMaxDynamicSharedMemorySize, LDS_BYTES));
        init = true;
    }
    Params P;
    P.x = (const uint8_t*)d_in[0];
    P.w = (const uint8_t*)d_in[1];
    P.sx = (const float*)d_in[2];
    P.sw = (const float*)d_in[3];
    for (int p = 0; p < NR; ++p) {
        P.out[p] = (float*)d_out_all[p];
        P.ws[p] = (uint8_t*)d_ws_all[p];
    }
    P.rank = rank;
    fused_kernel<<<GRID, THREADS, LDS_BYTES, stream>>>(P);
    HIP_CHECK(hipGetLastError());
}
```

```cpp
#include <hip/hip_runtime.h>
#include <rccl/rccl.h>
#include <cstdio>
#include <cstdint>

#define HIP_CHECK(expr) do { hipError_t _e = (expr); if (_e != hipSuccess) { \
    fprintf(stderr, "HIP error %s at %s:%d\n", hipGetErrorString(_e), __FILE__, __LINE__); \
    return; } } while (0)

namespace {

typedef int v8i __attribute__((ext_vector_type(8)));
typedef float v4f __attribute__((ext_vector_type(4)));
typedef unsigned v4u __attribute__((ext_vector_type(4)));
typedef __attribute__((address_space(3))) volatile unsigned lds_u32;

constexpr int NR = 8;
constexpr int GM = 4096, GK = 4096, GN = 16384;
constexpr int KS = GK / NR;
constexpr int NS = GN / NR;
constexpr int MS = GM / NR;
constexpr int TM = 128, TN = 256, BK = 128;
constexpr int NMB = GM / TM;
constexpr int NNB = NS / TN;
constexpr int NTILES = NMB * NNB;
constexpr int GTN = 128;
constexpr int NGT = NTILES * 2;
constexpr int NKT = GK / BK;

constexpr int GRID = 256, THREADS = 512;
#define COMM_WGS 32
constexpr int NCOMM = COMM_WGS;
constexpr int NPOOL = GRID - NCOMM;

constexpr size_t XS_BYTES = (size_t)GM * GK;
constexpr size_t WT_BYTES = (size_t)NS * GK;
constexpr size_t STAGE_BYTES = XS_BYTES + WT_BYTES;
constexpr size_t OFF_Y = 2 * STAGE_BYTES;
constexpr size_t Y_BYTES = (size_t)NTILES * TM * TN * 4;
constexpr size_t OFF_CTL = OFF_Y + Y_BYTES;
constexpr size_t WS_NEED = OFF_CTL + 131072;
#define STAMP(idx) ((void)0)
constexpr int CW = 32;
constexpr int C_CALL = 0 * CW, C_TILE = 1 * CW, C_TMO = 2 * CW, C_YCNT = 3 * CW;
constexpr int C_XCNT = 8 * CW;
constexpr int C_WCNT = 264 * CW;
constexpr int C_YFLAG = 328 * CW;

constexpr int LW = NCOMM / 8;
constexpr int XU = 4;
constexpr int U_W0 = 8, U_X0 = U_W0 + 8 * XU, U_W1 = U_X0 + 56, U_XW = U_W1 + 24 * XU;
constexpr int NYU = 32 * 64;
constexpr unsigned XCNT_PER_CALL = XU;
constexpr unsigned WCNT_PER_CALL = 8;
constexpr unsigned YCNT_PER_CALL = (NR - 1) * NCOMM * 7;

__device__ __forceinline__ int tset(int k) { return k; }

constexpr int STAGE_LDS = (GTN + TM) * BK;
constexpr int NSTG = 4;
constexpr int LDS_CTL = NSTG * STAGE_LDS;
constexpr int LDS_BYTES = LDS_CTL + 64;

constexpr unsigned long long DEADLINE_TICKS = 50000000ull;

struct Params {
    const uint8_t* x;
    const uint8_t* w;
    const float* sx;
    const float* sw;
    float* out[NR];
    uint8_t* ws[NR];
    int rank;
};

#define PSLEEP 32
template <bool SYS, int SLP = 2>
__device__ __forceinline__ void wait_ge(unsigned* p, unsigned target, unsigned* tmo, unsigned call) {
    unsigned long long t0 = __builtin_amdgcn_s_memrealtime();
    for (;;) {
        unsigned v = SYS ? __hip_atomic_load(p, __ATOMIC_RELAXED, __HIP_MEMORY_SCOPE_SYSTEM)
                         : __hip_atomic_load(p, __ATOMIC_RELAXED, __HIP_MEMORY_SCOPE_AGENT);
        if (v >= target) break;
        if (__hip_atomic_load(tmo, __ATOMIC_RELAXED, __HIP_MEMORY_SCOPE_AGENT) == call ||
            __builtin_amdgcn_s_memrealtime() - t0 > DEADLINE_TICKS) {
            __hip_atomic_store(tmo, call, __ATOMIC_RELAXED, __HIP_MEMORY_SCOPE_AGENT);
            break;
        }
        __builtin_amdgcn_s_sleep(SLP);
    }
}

__device__ __forceinline__ void tr4x4(unsigned r0, unsigned r1, unsigned r2, unsigned r3,
                                      unsigned& o0, unsigned& o1, unsigned& o2, unsigned& o3) {
    unsigned t0 = __builtin_amdgcn_perm(r1, r0, 0x05010400u);
    unsigned t1 = __builtin_amdgcn_perm(r1, r0, 0x07030602u);
    unsigned t2 = __builtin_amdgcn_perm(r3, r2, 0x05010400u);
    unsigned t3 = __builtin_amdgcn_perm(r3, r2, 0x07030602u);
    o0 = __builtin_amdgcn_perm(t2, t0, 0x05040100u);
    o1 = __builtin_amdgcn_perm(t2, t0, 0x07060302u);
    o2 = __builtin_amdgcn_perm(t3, t1, 0x05040100u);
    o3 = __builtin_amdgcn_perm(t3, t1, 0x07060302u);
}

#define VMCNT0() asm volatile("s_waitcnt vmcnt(0)" ::: "memory")
__device__ __forceinline__ void aload(v4u& d, const uint8_t* p) {
    asm volatile("global_load_dwordx4 %0, %1, off" : "=v"(d) : "v"(p) : "memory");
}
#define TIE4(a, i) "+v"(a[i]), "+v"(a[i + 1]), "+v"(a[i + 2]), "+v"(a[i + 3])
#define WAIT16(n, a) asm volatile("s_waitcnt vmcnt(" #n ")" : TIE4(a, 0), TIE4(a, 4), TIE4(a, 8), TIE4(a, 12) :: "memory")
#define WAIT14(n, a) asm volatile("s_waitcnt vmcnt(" #n ")" : TIE4(a, 0), TIE4(a, 4), TIE4(a, 8), "+v"(a[12]), "+v"(a[13]) :: "memory")

__device__ __forceinline__ int y_grab(lds_u32* sctl, int lane) {
    unsigned j = 0;
    if (lane == 0) j = __hip_atomic_fetch_add((__attribute__((address_space(3))) unsigned*)&sctl[4], 1u,
                                              __ATOMIC_RELAXED, __HIP_MEMORY_SCOPE_WORKGROUP);
    return __builtin_amdgcn_readfirstlane((int)j);
}
__device__ __forceinline__ bool y_ready(int wg, int j, lds_u32* sctl) {
    return (unsigned)__builtin_amdgcn_readfirstlane(sctl[2]) > (unsigned)((j * NCOMM + wg) >> 6);
}
__device__ __forceinline__ void y_load(v4u (&in)[14], const uint8_t* ybase, int rank, int wg, int j, int lane) {
    const int yu = j * NCOMM + wg;
    const int k = yu >> 6, sub = yu & 63;
    const uint8_t* src = ybase + (size_t)tset(k) * (TM * TN * 4) + sub * 2048 + lane * 16;
#pragma unroll
    for (int h = 0; h < 2; ++h)
#pragma unroll
        for (int q = 0; q < 7; ++q) {
            const int p = (rank + 1 + q) & 7;
            aload(in[h * 7 + q], src + (size_t)p * 32 * (TM * TN * 4) + h * 1024);
        }
}
__device__ __forceinline__ void y_store(v4u (&in)[14], const Params& P, int rank, int wg, int j, int lane,
                                        lds_u32* sctl) {
    const int yu = j * NCOMM + wg;
    const int k = yu >> 6, sub = yu & 63;
    const int tl = tset(k);
    const int rnd = tl >> 3, nb = tl & 7;
    const bool bad = __builtin_amdgcn_readfirstlane(sctl[3]) != 0;
    const v4u nanv = {0x7fc00000u, 0x7fc00000u, 0x7fc00000u, 0x7fc00000u};
    const int voff = ((rnd * TM + sub * 2) * GN + rank * NS + nb * TN) * 4 + lane * 16;
#pragma unroll
    for (int h = 0; h < 2; ++h)
#pragma unroll
        for (int q = 0; q < 7; ++q) {
            const int p = (rank + 1 + q) & 7;
            auto rs = __builtin_amdgcn_make_buffer_rsrc(P.out[p], 0, (int)((size_t)MS * GN * 4), 0x00020000);
            __builtin_amdgcn_raw_buffer_store_b128(bad ? nanv : in[h * 7 + q], rs, voff + h * (GN * 4), 0, 17);
        }
}

__device__ __forceinline__ void comm_role(const Params& P, unsigned call, int wg, int wave, int lane,
                                          lds_u32* sctl) {
    const int rank = P.rank;
    uint8_t* myws = P.ws[rank];
    unsigned* myctl = (unsigned*)(myws + OFF_CTL);
    unsigned* tmo = myctl + C_TMO;
    const size_t par = (size_t)(call & 1u) * STAGE_BYTES;
    unsigned* lctl;
    {
        uint8_t* q = P.ws[0];
#pragma unroll
        for (int i = 1; i < NR; ++i) q = (((rank + 1 + lane) & 7) == i) ? P.ws[i] : q;
        lctl = (unsigned*)(q + OFF_CTL);
    }
    if (wave == 7) {
        for (int k = 0; k < 32; ++k) {
            if (lane < 14) wait_ge<false>(myctl + C_YFLAG + ((((rank + 1 + (lane >> 1)) & 7) * 32 + tset(k)) * 2 + (lane & 1)) * CW, call, tmo, call);
            __builtin_amdgcn_fence(__ATOMIC_ACQUIRE, "agent");
            const bool bad = __hip_atomic_load(tmo, __ATOMIC_RELAXED, __HIP_MEMORY_SCOPE_AGENT) == call;
            VMCNT0();
            if (lane == 0) { if (bad) sctl[3] = 1u; sctl[2] = (unsigned)k + 1u; }
        }
        if (wg == 0 && lane == 0) {
            wait_ge<true>(myctl + C_YCNT, call * YCNT_PER_CALL, tmo, call);
            __builtin_amdgcn_fence(__ATOMIC_ACQUIRE, "");
            STAMP(1536);
            if (__hip_atomic_load(tmo, __ATOMIC_RELAXED, __HIP_MEMORY_SCOPE_AGENT) == call)
                P.out[rank][0] = __builtin_nanf("");
        }
        return;
    }
    const int JEND = (NYU - wg + NCOMM - 1) / NCOMM;
    const uint8_t* ybase = myws + OFF_Y;
    v4u A[14], B[14];
    int j = 0;
    bool pre = false;
    {
        const int p = wg & 7;
        unsigned* dctl = (unsigned*)(P.ws[p] + OFF_CTL);
        auto rs = __builtin_amdgcn_make_buffer_rsrc(P.ws[p] + par, 0, (int)STAGE_BYTES, 0x00020000);
        for (int u = (wg >> 3) * 7 + wave; u < U_XW; u += 7 * LW) {
            unsigned* cnt;
            if (u < U_W0 || (u >= U_X0 && u < U_W1)) {
                const int wu = (u < U_W0) ? u : (u - U_X0 + 8);
                const int nb = wu >> 3, kb = (wu >> 1) & 3, nh = wu & 1;
                const int kg = lane & 7, ng = lane >> 3;
                const uint8_t* src = P.w + (size_t)(kb * 128 + kg * 16) * GN + (p * NS + nb * TN + nh * 128 + ng * 16);
                v4u in[16];
#pragma unroll
                for (int i = 0; i < 16; ++i) in[i] = *(const v4u*)(src + (size_t)i * GN);
                v4u o[16];
#pragma unroll
                for (int a = 0; a < 4; ++a)
#pragma unroll
                    for (int b = 0; b < 4; ++b) {
                        unsigned q0, q1, q2, q3;
                        tr4x4(in[4 * a + 0][b], in[4 * a + 1][b], in[4 * a + 2][b], in[4 * a + 3][b], q0, q1, q2, q3);
                        o[4 * b + 0][a] = q0; o[4 * b + 1][a] = q1; o[4 * b + 2][a] = q2; o[4 * b + 3][a] = q3;
                    }
                const int voff = (int)XS_BYTES + rank * (NS * KS) + (nb * TN + nh * 128 + ng * 16) * KS + kb * 128 + kg * 16;
#pragma unroll
                for (int j = 0; j < 16; ++j)
                    __builtin_amdgcn_raw_buffer_store_b128(o[j], rs, voff + j * KS, 0, 17);
                cnt = dctl + C_WCNT + (nb * 8 + rank) * CW;
            } else {
                const int xu = (u < U_X0) ? (u - U_W0) : (u - U_W1 + 8 * XU);
                const int blk = xu / XU, sub = xu % XU;
                const int mb = ((p + 1 + (blk & 7)) & 7) * 4 + (blk >> 3);
                const int off = (mb * TM + sub * 32) * KS + lane * 16;
                v4u in[16];
#pragma unroll
                for (int i = 0; i < 16; ++i) in[i] = *(const v4u*)(P.x + off + i * 1024);
#pragma unroll
                for (int i = 0; i < 16; ++i)
                    __builtin_amdgcn_raw_buffer_store_b128(in[i], rs, rank * (GM * KS) + off + i * 1024, 0, 17);
                cnt = dctl + C_XCNT + (mb * 8 + rank) * CW;
            }
            if (u + 7 * LW >= U_XW) {
                j = y_grab(sctl, lane);
                pre = j < JEND && y_ready(wg, j, sctl);
                asm volatile("" ::: "memory");
                if (pre) y_load(A, ybase, rank, wg, j, lane);
                WAIT14(0, A);
            } else {
                VMCNT0();
            }
            if (lane == 0) { __hip_atomic_fetch_add(cnt, 1u, __ATOMIC_RELAXED, __HIP_MEMORY_SCOPE_SYSTEM); STAMP(2048 + p * 256 + u); }
        }
    }
    {
        if (j < JEND) {
            if (!pre) {
                while (!y_ready(wg, j, sctl)) __builtin_amdgcn_s_sleep(1);
                asm volatile("" ::: "memory");
                y_load(A, ybase, rank, wg, j, lane);
                WAIT14(0, A);
            }
#define Y_HALF(CUR, NXT)                                                       \
            {                                                                  \
                const int j1 = y_grab(sctl, lane);                             \
                const bool more = j1 < JEND;                                   \
                const bool rdy = more && y_ready(wg, j1, sctl);                \
                asm volatile("" ::: "memory");                                 \
                if (rdy) y_load(NXT, ybase, rank, wg, j1, lane);               \
                y_store(CUR, P, rank, wg, j, lane, sctl);                      \
                if (lane == 0) STAMP(4096 + wg * 64 + j);                      \
                if (!more) break;                                              \
                if (rdy) { WAIT14(14, NXT); }                                  \
                else {                                                         \
                    while (!y_ready(wg, j1, sctl)) __builtin_amdgcn_s_sleep(1);\
                    asm volatile("" ::: "memory");                             \
                    y_load(NXT, ybase, rank, wg, j1, lane);                    \
                    WAIT14(0, NXT);                                            \
                }                                                              \
                j = j1;                                                        \
            }
            for (;;) {
                Y_HALF(A, B)
                Y_HALF(B, A)
            }
        }
        VMCNT0();
        if (lane < 7) __hip_atomic_fetch_add(lctl + C_YCNT, 1u, __ATOMIC_RELAXED, __HIP_MEMORY_SCOPE_SYSTEM);
    }
}

__global__ __launch_bounds__(THREADS, 2) void fused_kernel(Params P) {
    extern __shared__ __attribute__((aligned(16))) uint8_t smem[];
    const int tid = threadIdx.x;
    const int wave = __builtin_amdgcn_readfirstlane(tid >> 6);
    const int lane = tid & 63;
    const int rank = P.rank;
    uint8_t* myws = P.ws[rank];
    unsigned* myctl = (unsigned*)(myws + OFF_CTL);
    unsigned* tmo = myctl + C_TMO;
    lds_u32* sctl = (lds_u32*)(smem + LDS_CTL);

    if (tid == 0) {
        unsigned t = __hip_atomic_fetch_add(myctl + C_CALL, 1u, __ATOMIC_RELAXED, __HIP_MEMORY_SCOPE_AGENT);
        sctl[0] = t / GRID + 1u;
        sctl[2] = 0u;
        sctl[3] = 0u;
        sctl[4] = 0u;
        STAMP(blockIdx.x);
    }
    __syncthreads();
    const unsigned call = __builtin_amdgcn_readfirstlane(sctl[0]);

    const int wg = blockIdx.x;
    if (wg < NCOMM) {
        comm_role(P, call, wg, wave, lane, sctl);
        return;
    }

    const float scale = P.sx[0] * P.sw[0];
    const uint8_t* stage = myws + (size_t)(call & 1u) * STAGE_BYTES;
    const int wn = wave >> 2, wm = wave & 3;
    const int fr = lane & 15, fq = lane >> 4;
    const int sw0 = ((fq) ^ (fr & 7)) * 16;
    const int a_off = (wn * 64 + fr) * BK + sw0;
    const int b_off = GTN * BK + (wm * 32 + fr) * BK + sw0;
    const int srow = lane >> 3;
    const int schunk = ((lane & 7) ^ (srow & 7)) * 16;

    for (;;) {
        __syncthreads();
        if (wave == 0) {
            unsigned t = 0;
            if (lane == 0) {
                t = __hip_atomic_fetch_add(myctl + C_TILE, 1u, __ATOMIC_RELAXED, __HIP_MEMORY_SCOPE_AGENT);
                t -= (call - 1u) * (unsigned)(NGT + NPOOL);
                sctl[1] = t;
            }
            t = __builtin_amdgcn_readfirstlane(t);
            if (t < (unsigned)NGT) {
                const int t2 = t >> 1;
                const int nb = (t2 < 64) ? (t2 >> 3) : (t2 & 7), oi = (t2 < 64) ? (t2 & 7) : ((t2 >> 3) & 7);
                const int mb = ((rank + 1 + oi) & 7) * 4 + (t2 >> 6);
                if (lane == 0) STAMP(256 + t2 * 4);
                if (lane < 16)
                    wait_ge<true, PSLEEP>(myctl + ((lane < 8) ? C_XCNT + (mb * 8 + lane) * CW : C_WCNT + (nb * 8 + lane - 8) * CW),
                                  call * ((lane < 8) ? XCNT_PER_CALL : WCNT_PER_CALL), tmo, call);
                __builtin_amdgcn_fence(__ATOMIC_ACQUIRE, "");
                VMCNT0();
                if (lane == 0) STAMP(256 + t2 * 4 + 1);
            }
        }
        __syncthreads();
        const unsigned t = __builtin_amdgcn_readfirstlane(sctl[1]);
        if (t >= (unsigned)NGT) break;
        const int t2 = t >> 1, half = t & 1;
        const int rnd = t2 >> 6, nb = (t2 < 64) ? (t2 >> 3) : (t2 & 7), oi = (t2 < 64) ? (t2 & 7) : ((t2 >> 3) & 7);
        const int owner = (rank + 1 + oi) & 7;
        const int mb = owner * 4 + rnd;

        const uint8_t* gA = stage + XS_BYTES + (size_t)(nb * TN + half * GTN + wave * 16 + srow) * KS + schunk;
        const uint8_t* gB = stage + (size_t)(mb * TM + wave * 16 + srow) * KS + schunk;
        auto issue = [&](int kt, int buf) {
            uint8_t* la = smem + buf + wave * 2048;
            uint8_t* lb = smem + buf + GTN * BK + wave * 2048;
            const uint8_t* a = gA + (size_t)(kt >> 2) * (NS * KS) + (kt & 3) * BK;
            const uint8_t* b = gB + (size_t)(kt >> 2) * (GM * KS) + (kt & 3) * BK;
#pragma unroll
            for (int j = 0; j < 2; ++j)
                __builtin_amdgcn_global_load_lds((const unsigned*)(a + j * 8 * KS), (unsigned*)(la + j * 1024), 16, 0, 0);
#pragma unroll
            for (int j = 0; j < 2; ++j)
                __builtin_amdgcn_global_load_lds((const unsigned*)(b + j * 8 * KS), (unsigned*)(lb + j * 1024), 16, 0, 0);
        };

        v4f acc[4][2];
#pragma unroll
        for (int i = 0; i < 4; ++i)
#pragma unroll
            for (int j = 0; j < 2; ++j) acc[i][j] = (v4f){0.f, 0.f, 0.f, 0.f};

        issue(0, 0);
        issue(1, STAGE_LDS);
        issue(2, 2 * STAGE_LDS);
        for (int kt = 0; kt < NKT; ++kt) {
            if (kt < NKT - 2) asm volatile("s_waitcnt vmcnt(8)" ::: "memory");
            else if (kt == NKT - 2) asm volatile("s_waitcnt vmcnt(4)" ::: "memory");
            else asm volatile("s_waitcnt vmcnt(0)" ::: "memory");
            asm volatile("s_waitcnt lgkmcnt(0)" ::: "memory");
            __builtin_amdgcn_s_barrier();
            asm volatile("" ::: "memory");
            const int cur = (kt & 3) * STAGE_LDS;
            if (kt + 3 < NKT) issue(kt + 3, ((kt + 3) & 3) * STAGE_LDS);
            v8i af[4], bf[2];
#pragma unroll
            for (int i = 0; i < 4; ++i) {
                const v4u lo = *(const v4u*)(smem + cur + a_off + i * 2048);
                const v4u hi = *(const v4u*)(smem + cur + (a_off ^ 64) + i * 2048);
                af[i] = (v8i){(int)lo[0], (int)lo[1], (int)lo[2], (int)lo[3], (int)hi[0], (int)hi[1], (int)hi[2], (int)hi[3]};
            }
#pragma unroll
            for (int i = 0; i < 2; ++i) {
                const v4u lo = *(const v4u*)(smem + cur + b_off + i * 2048);
                const v4u hi = *(const v4u*)(smem + cur + (b_off ^ 64) + i * 2048);
                bf[i] = (v8i){(int)lo[0], (int)lo[1], (int)lo[2], (int)lo[3], (int)hi[0], (int)hi[1], (int)hi[2], (int)hi[3]};
            }
#pragma unroll
            for (int i = 0; i < 4; ++i)
#pragma unroll
                for (int j = 0; j < 2; ++j)
                    acc[i][j] = __builtin_amdgcn_mfma_scale_f32_16x16x128_f8f6f4(af[i], bf[j], acc[i][j], 0, 0, 0,
                                                                                 0x7F7F7F7F, 0, 0x7F7F7F7F);
        }

        const bool bad = __hip_atomic_load(tmo, __ATOMIC_RELAXED, __HIP_MEMORY_SCOPE_AGENT) == call;
        const float s = bad ? __builtin_nanf("") : scale;
        if (tid == 0) STAMP(256 + t2 * 4 + 2);
        if (owner == rank) {
            float* dst = P.out[rank] + (size_t)(rnd * TM + wm * 32 + fr) * GN + rank * NS + nb * TN + half * GTN + wn * 64 + fq * 4;
#pragma unroll
            for (int j = 0; j < 2; ++j)
#pragma unroll
                for (int i = 0; i < 4; ++i)
                    *(v4f*)(dst + (size_t)j * 16 * GN + i * 16) = acc[i][j] * s;
        } else {
            const int tile = mb * NNB + nb;
            auto rs_y = __builtin_amdgcn_make_buffer_rsrc(myws + OFF_Y + (size_t)tile * (TM * TN * 4), 0, TM * TN * 4, 0x00020000);
            const int voff = ((wm * 32 + fr) * TN + half * GTN + wn * 64 + fq * 4) * 4;
#pragma unroll
            for (int j = 0; j < 2; ++j)
#pragma unroll
                for (int i = 0; i < 4; ++i) {
                    v4f v = acc[i][j] * s;
                    __builtin_amdgcn_raw_buffer_store_b128(__builtin_bit_cast(v4u, v), rs_y, voff + (j * 16 * TN + i * 16) * 4, 0, 16);
                }
            VMCNT0();
            __syncthreads();
            if (tid == 0) {
                __hip_atomic_store(myctl + C_YFLAG + (tile * 2 + half) * CW, call, __ATOMIC_RELAXED, __HIP_MEMORY_SCOPE_AGENT);
                STAMP(256 + t2 * 4 + 3);
            }
        }
    }
}

}

extern "C" void kernel_launch(int rank, int nranks, ncclComm_t comm,
                              hipStream_t stream,
                              void* const* d_in, const int* in_sizes, int n_in,
                              void* d_out, int out_size,
                              const int* in_dtypes, int out_dtype,
                              void* const* d_out_all,
                              void* const* d_ws_all, size_t ws_bytes) {
    (void)comm; (void)in_sizes; (void)n_in; (void)in_dtypes; (void)out_dtype; (void)d_out;
    if (nranks != NR || (unsigned)rank >= (unsigned)NR || ws_bytes < WS_NEED || out_size != MS * GN) {
        fprintf(stderr, "unsupported configuration\n");
        return;
    }
    static bool init = false;
    if (!init) {
        HIP_CHECK(hipFuncSetAttribute((const void*)fused_kernel, hipFuncAttributeMaxDynamicSharedMemorySize, LDS_BYTES));
        init = true;
    }
    Params P;
    P.x = (const uint8_t*)d_in[0];
    P.w = (const uint8_t*)d_in[1];
    P.sx = (const float*)d_in[2];
    P.sw = (const float*)d_in[3];
    for (int p = 0; p < NR; ++p) {
        P.out[p] = (float*)d_out_all[p];
        P.ws[p] = (uint8_t*)d_ws_all[p];
    }
    P.rank = rank;
    fused_kernel<<<GRID, THREADS, LDS_BYTES, stream>>>(P);
    HIP_CHECK(hipGetLastError());
}
```

```cpp
#include <hip/hip_runtime.h>
#include <rccl/rccl.h>
#include <cstdio>
#include <cstdint>

#define HIP_CHECK(expr) do { hipError_t _e = (expr); if (_e != hipSuccess) { \
    fprintf(stderr, "HIP error %s at %s:%d\n", hipGetErrorString(_e), __FILE__, __LINE__); \
    return; } } while (0)

namespace {

typedef int v8i __attribute__((ext_vector_type(8)));
typedef float v4f __attribute__((ext_vector_type(4)));
typedef unsigned v4u __attribute__((ext_vector_type(4)));
typedef __attribute__((address_space(3))) volatile unsigned lds_u32;

constexpr int NR = 8;
constexpr int GM = 4096, GK = 4096, GN = 16384;
constexpr int KS = GK / NR;
constexpr int NS = GN / NR;
constexpr int MS = GM / NR;
constexpr int TM = 128, TN = 256, BK = 128;
constexpr int NMB = GM / TM;
constexpr int NNB = NS / TN;
constexpr int NTILES = NMB * NNB;
constexpr int GTN = 128;
constexpr int NGT = NTILES * 2;
constexpr int NKT = GK / BK;

constexpr int GRID = 256, THREADS = 512;
constexpr int NCOMM = 32;
constexpr int NPOOL = GRID - NCOMM;

constexpr size_t XS_BYTES = (size_t)GM * GK;
constexpr size_t WT_BYTES = (size_t)NS * GK;
constexpr size_t STAGE_BYTES = XS_BYTES + WT_BYTES;
constexpr size_t OFF_Y = 2 * STAGE_BYTES;
constexpr size_t Y_BYTES = (size_t)NTILES * TM * TN * 4;
constexpr size_t OFF_CTL = OFF_Y + Y_BYTES;
constexpr size_t WS_NEED = OFF_CTL + 262144;
constexpr int CW = 32;
constexpr int C_CALL = 0 * CW, C_TMO = 2 * CW, C_YCNT = 3 * CW;
constexpr int C_XCNT = 8 * CW;
constexpr int C_WCNT = 264 * CW;
constexpr int C_YFLAG = 328 * CW;
constexpr int C_TILEQ = 840 * CW;

constexpr int LW = NCOMM / 8;
constexpr int XU = 4;
constexpr int U_W0 = 8, U_X0 = U_W0 + 8 * XU, U_W1 = U_X0 + 56, U_XW = U_W1 + 24 * XU;
constexpr int NYU = 32 * 64;
constexpr unsigned XCNT_PER_CALL = XU;
constexpr unsigned WCNT_PER_CALL = 8;
constexpr int YW = 4;
constexpr unsigned YCNT_PER_CALL = (NR - 1) * NCOMM * YW;

constexpr int STAGE_LDS = (GTN + TM) * BK;
constexpr int NSTG = 4;
constexpr int LDS_CTL = NSTG * STAGE_LDS;
constexpr int LDS_BYTES = LDS_CTL + 64;

constexpr unsigned long long DEADLINE_TICKS = 50000000ull;

struct Params {
    const uint8_t* x;
    const uint8_t* w;
    const float* sx;
    const float* sw;
    float* out[NR];
    uint8_t* ws[NR];
    int rank;
};

template <bool SYS, int SLP = 2>
__device__ __forceinline__ void wait_ge(unsigned* p, unsigned target, unsigned* tmo, unsigned call) {
    unsigned long long t0 = __builtin_amdgcn_s_memrealtime();
    for (;;) {
        unsigned v = SYS ? __hip_atomic_load(p, __ATOMIC_RELAXED, __HIP_MEMORY_SCOPE_SYSTEM)
                         : __hip_atomic_load(p, __ATOMIC_RELAXED, __HIP_MEMORY_SCOPE_AGENT);
        if (v >= target) break;
        if (__hip_atomic_load(tmo, __ATOMIC_RELAXED, __HIP_MEMORY_SCOPE_AGENT) == call ||
            __builtin_amdgcn_s_memrealtime() - t0 > DEADLINE_TICKS) {
            __hip_atomic_store(tmo, call, __ATOMIC_RELAXED, __HIP_MEMORY_SCOPE_AGENT);
            break;
        }
        __builtin_amdgcn_s_sleep(SLP);
    }
}

__device__ __forceinline__ void tr4x4(unsigned r0, unsigned r1, unsigned r2, unsigned r3,
                                      unsigned& o0, unsigned& o1, unsigned& o2, unsigned& o3) {
    unsigned t0 = __builtin_amdgcn_perm(r1, r0, 0x05010400u);
    unsigned t1 = __builtin_amdgcn_perm(r1, r0, 0x07030602u);
    unsigned t2 = __builtin_amdgcn_perm(r3, r2, 0x05010400u);
    unsigned t3 = __builtin_amdgcn_perm(r3, r2, 0x07030602u);
    o0 = __builtin_amdgcn_perm(t2, t0, 0x05040100u);
    o1 = __builtin_amdgcn_perm(t2, t0, 0x07060302u);
    o2 = __builtin_amdgcn_perm(t3, t1, 0x05040100u);
    o3 = __builtin_amdgcn_perm(t3, t1, 0x07060302u);
}

#define VMCNT0() asm volatile("s_waitcnt vmcnt(0)" ::: "memory")
__device__ __forceinline__ void aload(v4u& d, const uint8_t* p) {
    asm volatile("global_load_dwordx4 %0, %1, off" : "=v"(d) : "v"(p) : "memory");
}
#define TIE4(a, i) "+v"(a[i]), "+v"(a[i + 1]), "+v"(a[i + 2]), "+v"(a[i + 3])
#define WAIT16(n, a) asm volatile("s_waitcnt vmcnt(" #n ")" : TIE4(a, 0), TIE4(a, 4), TIE4(a, 8), TIE4(a, 12) :: "memory")
#define WAIT14(n, a) asm volatile("s_waitcnt vmcnt(" #n ")" : TIE4(a, 0), TIE4(a, 4), TIE4(a, 8), "+v"(a[12]), "+v"(a[13]) :: "memory")

__device__ __forceinline__ int y_grab(lds_u32* sctl, int lane) {
    unsigned j = 0;
    if (lane == 0) j = __hip_atomic_fetch_add((__attribute__((address_space(3))) unsigned*)&sctl[4], 1u,
                                              __ATOMIC_RELAXED, __HIP_MEMORY_SCOPE_WORKGROUP);
    return __builtin_amdgcn_readfirstlane((int)j);
}
__device__ __forceinline__ bool y_ready(int wg, int j, lds_u32* sctl) {
    return (unsigned)__builtin_amdgcn_readfirstlane(sctl[2]) > (unsigned)((j * NCOMM + wg) >> 6);
}
__device__ __forceinline__ void y_load(v4u (&in)[14], const uint8_t* ybase, int rank, int wg, int j, int lane) {
    const int yu = j * NCOMM + wg;
    const int k = yu >> 6, sub = yu & 63;
    const uint8_t* src = ybase + (size_t)k * (TM * TN * 4) + sub * 2048 + lane * 16;
#pragma unroll
    for (int h = 0; h < 2; ++h)
#pragma unroll
        for (int q = 0; q < 7; ++q) {
            const int p = (rank + 1 + q) & 7;
            aload(in[h * 7 + q], src + (size_t)p * 32 * (TM * TN * 4) + h * 1024);
        }
}
__device__ __forceinline__ void y_store(v4u (&in)[14], const Params& P, int rank, int wg, int j, int lane,
                                        lds_u32* sctl) {
    const int yu = j * NCOMM + wg;
    const int k = yu >> 6, sub = yu & 63;
    const int rnd = k >> 3, nb = k & 7;
    const bool bad = __builtin_amdgcn_readfirstlane(sctl[3]) != 0;
    const v4u nanv = {0x7fc00000u, 0x7fc00000u, 0x7fc00000u, 0x7fc00000u};
    const int voff = ((rnd * TM + sub * 2) * GN + rank * NS + nb * TN) * 4 + lane * 16;
#pragma unroll
    for (int h = 0; h < 2; ++h)
#pragma unroll
        for (int q = 0; q < 7; ++q) {
            const int p = (rank + 1 + q) & 7;
            auto rs = __builtin_amdgcn_make_buffer_rsrc(P.out[p], 0, (int)((size_t)MS * GN * 4), 0x00020000);
            __builtin_amdgcn_raw_buffer_store_b128(bad ? nanv : in[h * 7 + q], rs, voff + h * (GN * 4), 0, 17);
        }
}

__device__ __forceinline__ void comm_role(const Params& P, unsigned call, int wg, int wave, int lane,
                                          lds_u32* sctl) {
    const int rank = P.rank;
    uint8_t* myws = P.ws[rank];
    unsigned* myctl = (unsigned*)(myws + OFF_CTL);
    unsigned* tmo = myctl + C_TMO;
    const size_t par = (size_t)(call & 1u) * STAGE_BYTES;
    unsigned* lctl;
    {
        uint8_t* q = P.ws[0];
#pragma unroll
        for (int i = 1; i < NR; ++i) q = (((rank + 1 + lane) & 7) == i) ? P.ws[i] : q;
        lctl = (unsigned*)(q + OFF_CTL);
    }
    if (wave == 7) {
        for (int k = 0; k < 32; ++k) {
            if (lane < 14) wait_ge<false>(myctl + C_YFLAG + ((((rank + 1 + (lane >> 1)) & 7) * 32 + k) * 2 + (lane & 1)) * CW, call, tmo, call);
            __builtin_amdgcn_fence(__ATOMIC_ACQUIRE, "agent");
            const bool bad = __hip_atomic_load(tmo, __ATOMIC_RELAXED, __HIP_MEMORY_SCOPE_AGENT) == call;
            VMCNT0();
            if (lane == 0) { if (bad) sctl[3] = 1u; sctl[2] = (unsigned)k + 1u; }
        }
        if (wg == 0 && lane == 0) {
            wait_ge<true>(myctl + C_YCNT, call * YCNT_PER_CALL, tmo, call);
            __builtin_amdgcn_fence(__ATOMIC_ACQUIRE, "");
            if (__hip_atomic_load(tmo, __ATOMIC_RELAXED, __HIP_MEMORY_SCOPE_AGENT) == call)
                P.out[rank][0] = __builtin_nanf("");
        }
        return;
    }
    const int JEND = (NYU - wg + NCOMM - 1) / NCOMM;
    const uint8_t* ybase = myws + OFF_Y;
    v4u A[14], B[14];
    int j = 0;
    bool pre = false;
    {
        const int p = wg & 7;
        unsigned* dctl = (unsigned*)(P.ws[p] + OFF_CTL);
        auto rs = __builtin_amdgcn_make_buffer_rsrc(P.ws[p] + par, 0, (int)STAGE_BYTES, 0x00020000);
        for (int u = (wg >> 3) * 7 + wave; u < U_XW; u += 7 * LW) {
            unsigned* cnt;
            if (u < U_W0 || (u >= U_X0 && u < U_W1)) {
                const int wu = (u < U_W0) ? u : (u - U_X0 + 8);
                const int nb = wu >> 3, kb = (wu >> 1) & 3, nh = wu & 1;
                const int kg = lane & 7, ng = lane >> 3;
                const uint8_t* src = P.w + (size_t)(kb * 128 + kg * 16) * GN + (p * NS + nb * TN + nh * 128 + ng * 16);
                v4u in[16];
#pragma unroll
                for (int i = 0; i < 16; ++i) aload(in[i], src + (size_t)i * GN);
                WAIT16(0, in);
                v4u o[16];
#pragma unroll
                for (int a = 0; a < 4; ++a)
#pragma unroll
                    for (int b = 0; b < 4; ++b) {
                        unsigned q0, q1, q2, q3;
                        tr4x4(in[4 * a + 0][b], in[4 * a + 1][b], in[4 * a + 2][b], in[4 * a + 3][b], q0, q1, q2, q3);
                        o[4 * b + 0][a] = q0; o[4 * b + 1][a] = q1; o[4 * b + 2][a] = q2; o[4 * b + 3][a] = q3;
                    }
                const int voff = (int)XS_BYTES + rank * (NS * KS) + (nb * TN + nh * 128 + ng * 16) * KS + kb * 128 + kg * 16;
#pragma unroll
                for (int j = 0; j < 16; ++j)
                    __builtin_amdgcn_raw_buffer_store_b128(o[j], rs, voff + j * KS, 0, 17);
                cnt = dctl + C_WCNT + (nb * 8 + rank) * CW;
            } else {
                const int xu = (u < U_X0) ? (u - U_W0) : (u - U_W1 + 8 * XU);
                const int blk = xu / XU, sub = xu % XU;
                const int mb = ((p + 1 + (blk & 7)) & 7) * 4 + (blk >> 3);
                const int off = (mb * TM + sub * 32) * KS + lane * 16;
                v4u in[16];
#pragma unroll
                for (int i = 0; i < 16; ++i) in[i] = *(const v4u*)(P.x + off + i * 1024);
#pragma unroll
                for (int i = 0; i < 16; ++i)
                    __builtin_amdgcn_raw_buffer_store_b128(in[i], rs, rank * (GM * KS) + off + i * 1024, 0, 17);
                cnt = dctl + C_XCNT + (mb * 8 + rank) * CW;
            }
            if (u + 7 * LW >= U_XW && wave < YW) {
                j = y_grab(sctl, lane);
                pre = j < JEND && y_ready(wg, j, sctl);
                asm volatile("" ::: "memory");
                if (pre) y_load(A, ybase, rank, wg, j, lane);
                WAIT14(0, A);
            } else {
                VMCNT0();
            }
            if (lane == 0) __hip_atomic_fetch_add(cnt, 1u, __ATOMIC_RELAXED, __HIP_MEMORY_SCOPE_SYSTEM);
        }
    }
    if (wave < YW) {
        if (j < JEND) {
            if (!pre) {
                while (!y_ready(wg, j, sctl)) __builtin_amdgcn_s_sleep(1);
                asm volatile("" ::: "memory");
                y_load(A, ybase, rank, wg, j, lane);
                WAIT14(0, A);
            }
#define Y_HALF(CUR, NXT)                                                       \
            {                                                                  \
                const int j1 = y_grab(sctl, lane);                             \
                const bool more = j1 < JEND;                                   \
                const bool rdy = more && y_ready(wg, j1, sctl);                \
                asm volatile("" ::: "memory");                                 \
                if (rdy) y_load(NXT, ybase, rank, wg, j1, lane);               \
                y_store(CUR, P, rank, wg, j, lane, sctl);                      \
                if (!more) break;                                              \
                if (rdy) { WAIT14(14, NXT); }                                  \
                else {                                                         \
                    while (!y_ready(wg, j1, sctl)) __builtin_amdgcn_s_sleep(1);\
                    asm volatile("" ::: "memory");                             \
                    y_load(NXT, ybase, rank, wg, j1, lane);                    \
                    WAIT14(0, NXT);                                            \
                }                                                              \
                j = j1;                                                        \
            }
            for (;;) {
                Y_HALF(A, B)
                Y_HALF(B, A)
            }
        }
        VMCNT0();
        if (lane < 7) __hip_atomic_fetch_add(lctl + C_YCNT, 1u, __ATOMIC_RELAXED, __HIP_MEMORY_SCOPE_SYSTEM);
    }
}

__global__ __launch_bounds__(THREADS, 2) void fused_kernel(Params P) {
    extern __shared__ __attribute__((aligned(16))) uint8_t smem[];
    const int tid = threadIdx.x;
    const int wave = __builtin_amdgcn_readfirstlane(tid >> 6);
    const int lane = tid & 63;
    const int rank = P.rank;
    uint8_t* myws = P.ws[rank];
    unsigned* myctl = (unsigned*)(myws + OFF_CTL);
    unsigned* tmo = myctl + C_TMO;
    lds_u32* sctl = (lds_u32*)(smem + LDS_CTL);

    if (tid == 0) {
        unsigned t = __hip_atomic_fetch_add(myctl + C_CALL, 1u, __ATOMIC_RELAXED, __HIP_MEMORY_SCOPE_AGENT);
        sctl[0] = t / GRID + 1u;
        sctl[2] = 0u;
        sctl[3] = 0u;
        sctl[4] = 0u;
    }
    __syncthreads();
    const unsigned call = __builtin_amdgcn_readfirstlane(sctl[0]);

    const int wg = blockIdx.x;
    if (wg < NCOMM) {
        comm_role(P, call, wg, wave, lane, sctl);
        return;
    }

    const float scale = P.sx[0] * P.sw[0];
    const uint8_t* stage = myws + (size_t)(call & 1u) * STAGE_BYTES;
    const int wn = wave >> 2, wm = wave & 3;
    const int fr = lane & 15, fq = lane >> 4;
    const int sw0 = ((fq) ^ (fr & 7)) * 16;
    const int a_off = (wn * 64 + fr) * BK + sw0;
    const int b_off = GTN * BK + (wm * 32 + fr) * BK + sw0;
    const int srow = lane >> 3;
    const int schunk = ((lane & 7) ^ (srow & 7)) * 16;

    for (;;) {
        __syncthreads();
        if (wave == 0) {
            unsigned t = 0;
            if (lane == 0) {
                t = __hip_atomic_fetch_add(myctl + C_TILEQ + (wg & 7) * CW, 1u, __ATOMIC_RELAXED, __HIP_MEMORY_SCOPE_AGENT);
                t -= (call - 1u) * (unsigned)((NGT + NPOOL) / 8);
                if (t < (unsigned)(NGT / 8)) {
                    const unsigned rnd_ = t >> 4, oi_ = (t >> 1) & 7, nb_ = wg & 7;
                    const unsigned t2_ = rnd_ * 64 + (rnd_ == 0 ? nb_ * 8 + oi_ : oi_ * 8 + nb_);
                    t = t2_ * 2 + (t & 1);
                } else {
                    t = NGT;
                }
                sctl[1] = t;
            }
            t = __builtin_amdgcn_readfirstlane(t);
            if (t < (unsigned)NGT) {
                const int t2 = t >> 1;
                const int nb = (t2 < 64) ? (t2 >> 3) : (t2 & 7), oi = (t2 < 64) ? (t2 & 7) : ((t2 >> 3) & 7);
                const int mb = ((rank + 1 + oi) & 7) * 4 + (t2 >> 6);
                if (lane < 16)
                    wait_ge<true, 16>(myctl + ((lane < 8) ? C_XCNT + (mb * 8 + lane) * CW : C_WCNT + (nb * 8 + lane - 8) * CW),
                                  call * ((lane < 8) ? XCNT_PER_CALL : WCNT_PER_CALL), tmo, call);
                __builtin_amdgcn_fence(__ATOMIC_ACQUIRE, "");
                VMCNT0();
            }
        }
        __syncthreads();
        const unsigned t = __builtin_amdgcn_readfirstlane(sctl[1]);
        if (t >= (unsigned)NGT) break;
        const int t2 = t >> 1, half = t & 1;
        const int rnd = t2 >> 6, nb = (t2 < 64) ? (t2 >> 3) : (t2 & 7), oi = (t2 < 64) ? (t2 & 7) : ((t2 >> 3) & 7);
        const int owner = (rank + 1 + oi) & 7;
        const int mb = owner * 4 + rnd;

        const uint8_t* gA = stage + XS_BYTES + (size_t)(nb * TN + half * GTN + wave * 16 + srow) * KS + schunk;
        const uint8_t* gB = stage + (size_t)(mb * TM + wave * 16 + srow) * KS + schunk;
        auto issue = [&](int kt, int buf) {
            uint8_t* la = smem + buf + wave * 2048;
            uint8_t* lb = smem + buf + GTN * BK + wave * 2048;
            const uint8_t* a = gA + (size_t)(kt >> 2) * (NS * KS) + (kt & 3) * BK;
            const uint8_t* b = gB + (size_t)(kt >> 2) * (GM * KS) + (kt & 3) * BK;
#pragma unroll
            for (int j = 0; j < 2; ++j)
                __builtin_amdgcn_global_load_lds((const unsigned*)(a + j * 8 * KS), (unsigned*)(la + j * 1024), 16, 0, 0);
#pragma unroll
            for (int j = 0; j < 2; ++j)
                __builtin_amdgcn_global_load_lds((const unsigned*)(b + j * 8 * KS), (unsigned*)(lb + j * 1024), 16, 0, 0);
        };

        v4f acc[4][2];
#pragma unroll
        for (int i = 0; i < 4; ++i)
#pragma unroll
            for (int j = 0; j < 2; ++j) acc[i][j] = (v4f){0.f, 0.f, 0.f, 0.f};

        issue(0, 0);
        issue(1, STAGE_LDS);
        issue(2, 2 * STAGE_LDS);
        for (int kt = 0; kt < NKT; ++kt) {
            if (kt < NKT - 2) asm volatile("s_waitcnt vmcnt(8)" ::: "memory");
            else if (kt == NKT - 2) asm volatile("s_waitcnt vmcnt(4)" ::: "memory");
            else asm volatile("s_waitcnt vmcnt(0)" ::: "memory");
            asm volatile("s_waitcnt lgkmcnt(0)" ::: "memory");
            __builtin_amdgcn_s_barrier();
            asm volatile("" ::: "memory");
            const int cur = (kt & 3) * STAGE_LDS;
            if (kt + 3 < NKT) issue(kt + 3, ((kt + 3) & 3) * STAGE_LDS);
            v8i af[4], bf[2];
#pragma unroll
            for (int i = 0; i < 4; ++i) {
                const v4u lo = *(const v4u*)(smem + cur + a_off + i * 2048);
                const v4u hi = *(const v4u*)(smem + cur + (a_off ^ 64) + i * 2048);
                af[i] = (v8i){(int)lo[0], (int)lo[1], (int)lo[2], (int)lo[3], (int)hi[0], (int)hi[1], (int)hi[2], (int)hi[3]};
            }
#pragma unroll
            for (int i = 0; i < 2; ++i) {
                const v4u lo = *(const v4u*)(smem + cur + b_off + i * 2048);
                const v4u hi = *(const v4u*)(smem + cur + (b_off ^ 64) + i * 2048);
                bf[i] = (v8i){(int)lo[0], (int)lo[1], (int)lo[2], (int)lo[3], (int)hi[0], (int)hi[1], (int)hi[2], (int)hi[3]};
            }
#pragma unroll
            for (int i = 0; i < 4; ++i)
#pragma unroll
                for (int j = 0; j < 2; ++j)
                    acc[i][j] = __builtin_amdgcn_mfma_scale_f32_16x16x128_f8f6f4(af[i], bf[j], acc[i][j], 0, 0, 0,
                                                                                 0x7F7F7F7F, 0, 0x7F7F7F7F);
        }

        const bool bad = __hip_atomic_load(tmo, __ATOMIC_RELAXED, __HIP_MEMORY_SCOPE_AGENT) == call;
        const float s = bad ? __builtin_nanf("") : scale;
        if (owner == rank) {
            auto rs_o = __builtin_amdgcn_make_buffer_rsrc(P.out[rank], 0, (int)((size_t)MS * GN * 4), 0x00020000);
            const int voff = ((rnd * TM + wm * 32 + fr) * GN + rank * NS + nb * TN + half * GTN + wn * 64 + fq * 4) * 4;
#pragma unroll
            for (int j = 0; j < 2; ++j)
#pragma unroll
                for (int i = 0; i < 4; ++i) {
                    v4f v = acc[i][j] * s;
                    __builtin_amdgcn_raw_buffer_store_b128(__builtin_bit_cast(v4u, v), rs_o, voff + (j * 16 * GN + i * 16) * 4, 0, 16);
                }
        } else {
            const int tile = mb * NNB + nb;
            auto rs_y = __builtin_amdgcn_make_buffer_rsrc(myws + OFF_Y + (size_t)tile * (TM * TN * 4), 0, TM * TN * 4, 0x00020000);
            const int voff = ((wm * 32 + fr) * TN + half * GTN + wn * 64 + fq * 4) * 4;
#pragma unroll
            for (int j = 0; j < 2; ++j)
#pragma unroll
                for (int i = 0; i < 4; ++i) {
                    v4f v = acc[i][j] * s;
                    __builtin_amdgcn_raw_buffer_store_b128(__builtin_bit_cast(v4u, v), rs_y, voff + (j * 16 * TN + i * 16) * 4, 0, 16);
                }
            VMCNT0();
            __syncthreads();
            if (tid == 0)
                __hip_atomic_store(myctl + C_YFLAG + (tile * 2 + half) * CW, call, __ATOMIC_RELAXED, __HIP_MEMORY_SCOPE_AGENT);
        }
    }
}

}

extern "C" void kernel_launch(int rank, int nranks, ncclComm_t comm,
                              hipStream_t stream,
                              void* const* d_in, const int* in_sizes, int n_in,
                              void* d_out, int out_size,
                              const int* in_dtypes, int out_dtype,
                              void* const* d_out_all,
                              void* const* d_ws_all, size_t ws_bytes) {
    (void)comm; (void)in_sizes; (void)n_in; (void)in_dtypes; (void)out_dtype; (void)d_out;
    if (nranks != NR || (unsigned)rank >= (unsigned)NR || ws_bytes < WS_NEED || out_size != MS * GN) {
        fprintf(stderr, "unsupported configuration\n");
        return;
    }
    static bool init = false;
    if (!init) {
        HIP_CHECK(hipFuncSetAttribute((const void*)fused_kernel, hipFuncAttributeMaxDynamicSharedMemorySize, LDS_BYTES));
        init = true;
    }
    Params P;
    P.x = (const uint8_t*)d_in[0];
    P.w = (const uint8_t*)d_in[1];
    P.sx = (const float*)d_in[2];
    P.sw = (const float*)d_in[3];
    for (int p = 0; p < NR; ++p) {
        P.out[p] = (float*)d_out_all[p];
        P.ws[p] = (uint8_t*)d_ws_all[p];
    }
    P.rank = rank;
    fused_kernel<<<GRID, THREADS, LDS_BYTES, stream>>>(P);
    HIP_CHECK(hipGetLastError());
}
```

```cpp
#include <hip/hip_runtime.h>
#include <rccl/rccl.h>
#include <cstdio>
#include <cstdint>

#define HIP_CHECK(expr) do { hipError_t _e = (expr); if (_e != hipSuccess) { \
    fprintf(stderr, "HIP error %s at %s:%d\n", hipGetErrorString(_e), __FILE__, __LINE__); \
    return; } } while (0)

namespace {

typedef int v8i __attribute__((ext_vector_type(8)));
typedef float v4f __attribute__((ext_vector_type(4)));
typedef unsigned v4u __attribute__((ext_vector_type(4)));
typedef __attribute__((address_space(3))) volatile unsigned lds_u32;

constexpr int NR = 8;
constexpr int GM = 4096, GK = 4096, GN = 16384;
constexpr int KS = GK / NR;
constexpr int NS = GN / NR;
constexpr int MS = GM / NR;
constexpr int TM = 128, TN = 256, BK = 128;
constexpr int NMB = GM / TM;
constexpr int NNB = NS / TN;
constexpr int NTILES = NMB * NNB;
constexpr int GTN = 128;
constexpr int NGT = NTILES * 2;
constexpr int NKT = GK / BK;

constexpr int GRID = 256, THREADS = 512;
constexpr int NCOMM = 32;
constexpr int NPOOL = GRID - NCOMM;

constexpr size_t XS_BYTES = (size_t)GM * GK;
constexpr size_t WT_BYTES = (size_t)NS * GK;
constexpr size_t STAGE_BYTES = XS_BYTES + WT_BYTES;
constexpr size_t OFF_Y = 2 * STAGE_BYTES;
constexpr size_t Y_BYTES = (size_t)NTILES * TM * TN * 4;
constexpr size_t OFF_CTL = OFF_Y + Y_BYTES;
constexpr size_t WS_NEED = OFF_CTL + 262144;
constexpr int CW = 32;
constexpr int C_CALL = 0 * CW, C_TMO = 2 * CW;
constexpr int C_XCNT = 8 * CW;
constexpr int C_WCNT = 264 * CW;
constexpr int C_YFLAG = 328 * CW;
constexpr int C_TILEQ = 840 * CW;

constexpr int LW = NCOMM / 8;
constexpr int XU = 4;
constexpr int U_W0 = 8, U_X0 = U_W0 + 8 * XU, U_W1 = U_X0 + 56, U_XW = U_W1 + 24 * XU;
constexpr int NYU = 32 * 64;
constexpr unsigned XCNT_PER_CALL = XU;
constexpr unsigned WCNT_PER_CALL = 8;
constexpr int YW = 3;

constexpr int STAGE_LDS = (GTN + TM) * BK;
constexpr int NSTG = 4;
constexpr int LDS_CTL = NSTG * STAGE_LDS;
constexpr int LDS_BYTES = LDS_CTL + 64;

constexpr unsigned long long DEADLINE_TICKS = 50000000ull;

struct Params {
    const uint8_t* x;
    const uint8_t* w;
    const float* sx;
    const float* sw;
    float* out[NR];
    uint8_t* ws[NR];
    int rank;
};

template <bool SYS, int SLP = 2>
__device__ __forceinline__ void wait_ge(unsigned* p, unsigned target, unsigned* tmo, unsigned call) {
    unsigned long long t0 = __builtin_amdgcn_s_memrealtime();
    for (;;) {
        unsigned v = SYS ? __hip_atomic_load(p, __ATOMIC_RELAXED, __HIP_MEMORY_SCOPE_SYSTEM)
                         : __hip_atomic_load(p, __ATOMIC_RELAXED, __HIP_MEMORY_SCOPE_AGENT);
        if (v >= target) break;
        if (__hip_atomic_load(tmo, __ATOMIC_RELAXED, __HIP_MEMORY_SCOPE_AGENT) == call ||
            __builtin_amdgcn_s_memrealtime() - t0 > DEADLINE_TICKS) {
            __hip_atomic_store(tmo, call, __ATOMIC_RELAXED, __HIP_MEMORY_SCOPE_AGENT);
            break;
        }
        __builtin_amdgcn_s_sleep(SLP);
    }
}

__device__ __forceinline__ void tr4x4(unsigned r0, unsigned r1, unsigned r2, unsigned r3,
                                      unsigned& o0, unsigned& o1, unsigned& o2, unsigned& o3) {
    unsigned t0 = __builtin_amdgcn_perm(r1, r0, 0x05010400u);
    unsigned t1 = __builtin_amdgcn_perm(r1, r0, 0x07030602u);
    unsigned t2 = __builtin_amdgcn_perm(r3, r2, 0x05010400u);
    unsigned t3 = __builtin_amdgcn_perm(r3, r2, 0x07030602u);
    o0 = __builtin_amdgcn_perm(t2, t0, 0x05040100u);
    o1 = __builtin_amdgcn_perm(t2, t0, 0x07060302u);
    o2 = __builtin_amdgcn_perm(t3, t1, 0x05040100u);
    o3 = __builtin_amdgcn_perm(t3, t1, 0x07060302u);
}

#define VMCNT0() asm volatile("s_waitcnt vmcnt(0)" ::: "memory")
__device__ __forceinline__ void aload(v4u& d, const uint8_t* p) {
    asm volatile("global_load_dwordx4 %0, %1, off" : "=v"(d) : "v"(p) : "memory");
}
__device__ __forceinline__ void aload_sys(v4u& d, const uint8_t* p) {
    asm volatile("global_load_dwordx4 %0, %1, off sc0 sc1" : "=v"(d) : "v"(p) : "memory");
}
#define TIE4(a, i) "+v"(a[i]), "+v"(a[i + 1]), "+v"(a[i + 2]), "+v"(a[i + 3])
#define WAIT16(n, a) asm volatile("s_waitcnt vmcnt(" #n ")" : TIE4(a, 0), TIE4(a, 4), TIE4(a, 8), TIE4(a, 12) :: "memory")
#define WAIT14(n, a) asm volatile("s_waitcnt vmcnt(" #n ")" : TIE4(a, 0), TIE4(a, 4), TIE4(a, 8), "+v"(a[12]), "+v"(a[13]) :: "memory")

__device__ __forceinline__ int y_grab(lds_u32* sctl, int lane) {
    unsigned j = 0;
    if (lane == 0) j = __hip_atomic_fetch_add((__attribute__((address_space(3))) unsigned*)&sctl[4], 1u,
                                              __ATOMIC_RELAXED, __HIP_MEMORY_SCOPE_WORKGROUP);
    return __builtin_amdgcn_readfirstlane((int)j);
}
__device__ __forceinline__ bool y_ready(int wg, int j, lds_u32* sctl) {
    return (unsigned)__builtin_amdgcn_readfirstlane(sctl[2]) > (unsigned)((j * NCOMM + wg) >> 6);
}
__device__ __forceinline__ void y_load(v4u (&in)[14], const Params& P, int rank, int wg, int j, int lane) {
    const int yu = j * NCOMM + wg;
    const int k = yu >> 6, sub = yu & 63;
    const size_t off = OFF_Y + (size_t)(rank * 32 + k) * (TM * TN * 4) + sub * 2048 + lane * 16;
#pragma unroll
    for (int h = 0; h < 2; ++h)
#pragma unroll
        for (int q = 0; q < 7; ++q)
            aload_sys(in[h * 7 + q], P.ws[(rank + 1 + q) & 7] + off + h * 1024);
}
__device__ __forceinline__ void y_store(v4u (&in)[14], const Params& P, int rank, int wg, int j, int lane,
                                        lds_u32* sctl) {
    const int yu = j * NCOMM + wg;
    const int k = yu >> 6, sub = yu & 63;
    const int rnd = k >> 3, nb = k & 7;
    const bool bad = __builtin_amdgcn_readfirstlane(sctl[3]) != 0;
    const v4u nanv = {0x7fc00000u, 0x7fc00000u, 0x7fc00000u, 0x7fc00000u};
    auto rs = __builtin_amdgcn_make_buffer_rsrc(P.out[rank], 0, (int)((size_t)MS * GN * 4), 0x00020000);
    const int voff = ((rnd * TM + sub * 2) * GN + nb * TN) * 4 + lane * 16;
#pragma unroll
    for (int h = 0; h < 2; ++h)
#pragma unroll
        for (int q = 0; q < 7; ++q)
            __builtin_amdgcn_raw_buffer_store_b128(bad ? nanv : in[h * 7 + q], rs,
                                                   voff + h * (GN * 4) + ((rank + 1 + q) & 7) * (NS * 4), 0, 16);
}

__device__ __forceinline__ void comm_role(const Params& P, unsigned call, int wg, int wave, int lane,
                                          lds_u32* sctl) {
    const int rank = P.rank;
    uint8_t* myws = P.ws[rank];
    unsigned* myctl = (unsigned*)(myws + OFF_CTL);
    unsigned* tmo = myctl + C_TMO;
    const size_t par = (size_t)(call & 1u) * STAGE_BYTES;
    if (wave == 7) {
        for (int k = 0; k < 32; ++k) {
            if (lane < 14) wait_ge<true>(myctl + C_YFLAG + ((((rank + 1 + (lane >> 1)) & 7) * 32 + k) * 2 + (lane & 1)) * CW, call, tmo, call);
            __builtin_amdgcn_fence(__ATOMIC_ACQUIRE, "");
            const bool bad = __hip_atomic_load(tmo, __ATOMIC_RELAXED, __HIP_MEMORY_SCOPE_AGENT) == call;
            VMCNT0();
            if (lane == 0) { if (bad) sctl[3] = 1u; sctl[2] = (unsigned)k + 1u; }
        }
        return;
    }
    const int JEND = (NYU - wg + NCOMM - 1) / NCOMM;
    v4u A[14], B[14];
    int j = 0;
    bool pre = false;
    {
        const int p = wg & 7;
        unsigned* dctl = (unsigned*)(P.ws[p] + OFF_CTL);
        auto rs = __builtin_amdgcn_make_buffer_rsrc(P.ws[p] + par, 0, (int)STAGE_BYTES, 0x00020000);
        for (int u = (wg >> 3) * 7 + wave; u < U_XW; u += 7 * LW) {
            unsigned* cnt;
            if (u < U_W0 || (u >= U_X0 && u < U_W1)) {
                const int wu = (u < U_W0) ? u : (u - U_X0 + 8);
                const int nb = wu >> 3, kb = (wu >> 1) & 3, nh = wu & 1;
                const int kg = lane & 7, ng = lane >> 3;
                const uint8_t* src = P.w + (size_t)(kb * 128 + kg * 16) * GN + (p * NS + nb * TN + nh * 128 + ng * 16);
                v4u in[16];
#pragma unroll
                for (int i = 0; i < 16; ++i) aload(in[i], src + (size_t)i * GN);
                WAIT16(0, in);
                v4u o[16];
#pragma unroll
                for (int a = 0; a < 4; ++a)
#pragma unroll
                    for (int b = 0; b < 4; ++b) {
                        unsigned q0, q1, q2, q3;
                        tr4x4(in[4 * a + 0][b], in[4 * a + 1][b], in[4 * a + 2][b], in[4 * a + 3][b], q0, q1, q2, q3);
                        o[4 * b + 0][a] = q0; o[4 * b + 1][a] = q1; o[4 * b + 2][a] = q2; o[4 * b + 3][a] = q3;
                    }
                const int voff = (int)XS_BYTES + rank * (NS * KS) + (nb * TN + nh * 128 + ng * 16) * KS + kb * 128 + kg * 16;
#pragma unroll
                for (int j = 0; j < 16; ++j)
                    __builtin_amdgcn_raw_buffer_store_b128(o[j], rs, voff + j * KS, 0, 17);
                cnt = dctl + C_WCNT + (nb * 8 + rank) * CW;
            } else {
                const int xu = (u < U_X0) ? (u - U_W0) : (u - U_W1 + 8 * XU);
                const int blk = xu / XU, sub = xu % XU;
                const int mb = ((p + 1 + (blk & 7)) & 7) * 4 + (blk >> 3);
                const int off = (mb * TM + sub * 32) * KS + lane * 16;
                v4u in[16];
#pragma unroll
                for (int i = 0; i < 16; ++i) in[i] = *(const v4u*)(P.x + off + i * 1024);
#pragma unroll
                for (int i = 0; i < 16; ++i)
                    __builtin_amdgcn_raw_buffer_store_b128(in[i], rs, rank * (GM * KS) + off + i * 1024, 0, 17);
                cnt = dctl + C_XCNT + (mb * 8 + rank) * CW;
            }
            if (u + 7 * LW >= U_XW && wave < YW) {
                j = y_grab(sctl, lane);
                pre = j < JEND && y_ready(wg, j, sctl);
                asm volatile("" ::: "memory");
                if (pre) y_load(A, P, rank, wg, j, lane);
                WAIT14(0, A);
            } else {
                VMCNT0();
            }
            if (lane == 0) __hip_atomic_fetch_add(cnt, 1u, __ATOMIC_RELAXED, __HIP_MEMORY_SCOPE_SYSTEM);
        }
    }
    if (wave < YW) {
        if (j < JEND) {
            if (!pre) {
                while (!y_ready(wg, j, sctl)) __builtin_amdgcn_s_sleep(1);
                asm volatile("" ::: "memory");
                y_load(A, P, rank, wg, j, lane);
                WAIT14(0, A);
            }
#define Y_HALF(CUR, NXT)                                                       \
            {                                                                  \
                const int j1 = y_grab(sctl, lane);                             \
                const bool more = j1 < JEND;                                   \
                const bool rdy = more && y_ready(wg, j1, sctl);                \
                asm volatile("" ::: "memory");                                 \
                if (rdy) y_load(NXT, P, rank, wg, j1, lane);                   \
                y_store(CUR, P, rank, wg, j, lane, sctl);                      \
                if (!more) break;                                              \
                if (rdy) { WAIT14(14, NXT); }                                  \
                else {                                                         \
                    while (!y_ready(wg, j1, sctl)) __builtin_amdgcn_s_sleep(1);\
                    asm volatile("" ::: "memory");                             \
                    y_load(NXT, P, rank, wg, j1, lane);                        \
                    WAIT14(0, NXT);                                            \
                }                                                              \
                j = j1;                                                        \
            }
            for (;;) {
                Y_HALF(A, B)
                Y_HALF(B, A)
            }
        }
    }
}

__global__ __launch_bounds__(THREADS, 2) void fused_kernel(Params P) {
    extern __shared__ __attribute__((aligned(16))) uint8_t smem[];
    const int tid = threadIdx.x;
    const int wave = __builtin_amdgcn_readfirstlane(tid >> 6);
    const int lane = tid & 63;
    const int rank = P.rank;
    uint8_t* myws = P.ws[rank];
    unsigned* myctl = (unsigned*)(myws + OFF_CTL);
    unsigned* tmo = myctl + C_TMO;
    lds_u32* sctl = (lds_u32*)(smem + LDS_CTL);

    if (tid == 0) {
        unsigned t = __hip_atomic_fetch_add(myctl + C_CALL, 1u, __ATOMIC_RELAXED, __HIP_MEMORY_SCOPE_AGENT);
        sctl[0] = t / GRID + 1u;
        sctl[2] = 0u;
        sctl[3] = 0u;
        sctl[4] = 0u;
    }
    __syncthreads();
    const unsigned call = __builtin_amdgcn_readfirstlane(sctl[0]);

    const int wg = blockIdx.x;
    if (wg < NCOMM) {
        comm_role(P, call, wg, wave, lane, sctl);
        return;
    }

    const float scale = P.sx[0] * P.sw[0];
    const uint8_t* stage = myws + (size_t)(call & 1u) * STAGE_BYTES;
    const int wn = wave >> 2, wm = wave & 3;
    const int fr = lane & 15, fq = lane >> 4;
    const int sw0 = ((fq) ^ (fr & 7)) * 16;
    const int a_off = (wn * 64 + fr) * BK + sw0;
    const int b_off = GTN * BK + (wm * 32 + fr) * BK + sw0;
    const int srow = lane >> 3;
    const int schunk = ((lane & 7) ^ (srow & 7)) * 16;

    for (;;) {
        __syncthreads();
        if (wave == 0) {
            unsigned t = 0;
            if (lane == 0) {
                t = __hip_atomic_fetch_add(myctl + C_TILEQ + (wg & 7) * CW, 1u, __ATOMIC_RELAXED, __HIP_MEMORY_SCOPE_AGENT);
                t -= (call - 1u) * (unsigned)((NGT + NPOOL) / 8);
                if (t < (unsigned)(NGT / 8)) {
                    const unsigned rnd_ = t >> 4, oi_ = (t >> 1) & 7, nb_ = wg & 7;
                    const unsigned t2_ = rnd_ * 64 + (rnd_ == 0 ? nb_ * 8 + oi_ : oi_ * 8 + nb_);
                    t = t2_ * 2 + (t & 1);
                } else {
                    t = NGT;
                }
                sctl[1] = t;
            }
            t = __builtin_amdgcn_readfirstlane(t);
            if (t < (unsigned)NGT) {
                const int t2 = t >> 1;
                const int nb = (t2 < 64) ? (t2 >> 3) : (t2 & 7), oi = (t2 < 64) ? (t2 & 7) : ((t2 >> 3) & 7);
                const int mb = ((rank + 1 + oi) & 7) * 4 + (t2 >> 6);
                if (lane < 16)
                    wait_ge<true, 16>(myctl + ((lane < 8) ? C_XCNT + (mb * 8 + lane) * CW : C_WCNT + (nb * 8 + lane - 8) * CW),
                                  call * ((lane < 8) ? XCNT_PER_CALL : WCNT_PER_CALL), tmo, call);
                __builtin_amdgcn_fence(__ATOMIC_ACQUIRE, "");
                VMCNT0();
            }
        }
        __syncthreads();
        const unsigned t = __builtin_amdgcn_readfirstlane(sctl[1]);
        if (t >= (unsigned)NGT) break;
        const int t2 = t >> 1, half = t & 1;
        const int rnd = t2 >> 6, nb = (t2 < 64) ? (t2 >> 3) : (t2 & 7), oi = (t2 < 64) ? (t2 & 7) : ((t2 >> 3) & 7);
        const int owner = (rank + 1 + oi) & 7;
        const int mb = owner * 4 + rnd;

        const uint8_t* gA = stage + XS_BYTES + (size_t)(nb * TN + half * GTN + wave * 16 + srow) * KS + schunk;
        const uint8_t* gB = stage + (size_t)(mb * TM + wave * 16 + srow) * KS + schunk;
        auto issue = [&](int kt, int buf) {
            uint8_t* la = smem + buf + wave * 2048;
            uint8_t* lb = smem + buf + GTN * BK + wave * 2048;
            const uint8_t* a = gA + (size_t)(kt >> 2) * (NS * KS) + (kt & 3) * BK;
            const uint8_t* b = gB + (size_t)(kt >> 2) * (GM * KS) + (kt & 3) * BK;
#pragma unroll
            for (int j = 0; j < 2; ++j)
                __builtin_amdgcn_global_load_lds((const unsigned*)(a + j * 8 * KS), (unsigned*)(la + j * 1024), 16, 0, 0);
#pragma unroll
            for (int j = 0; j < 2; ++j)
                __builtin_amdgcn_global_load_lds((const unsigned*)(b + j * 8 * KS), (unsigned*)(lb + j * 1024), 16, 0, 0);
        };

        v4f acc[4][2];
#pragma unroll
        for (int i = 0; i < 4; ++i)
#pragma unroll
            for (int j = 0; j < 2; ++j) acc[i][j] = (v4f){0.f, 0.f, 0.f, 0.f};

        issue(0, 0);
        issue(1, STAGE_LDS);
        issue(2, 2 * STAGE_LDS);
        for (int kt = 0; kt < NKT; ++kt) {
            if (kt < NKT - 2) asm volatile("s_waitcnt vmcnt(8)" ::: "memory");
            else if (kt == NKT - 2) asm volatile("s_waitcnt vmcnt(4)" ::: "memory");
            else asm volatile("s_waitcnt vmcnt(0)" ::: "memory");
            asm volatile("s_waitcnt lgkmcnt(0)" ::: "memory");
            __builtin_amdgcn_s_barrier();
            asm volatile("" ::: "memory");
            const int cur = (kt & 3) * STAGE_LDS;
            if (kt + 3 < NKT) issue(kt + 3, ((kt + 3) & 3) * STAGE_LDS);
            v8i af[4], bf[2];
#pragma unroll
            for (int i = 0; i < 4; ++i) {
                const v4u lo = *(const v4u*)(smem + cur + a_off + i * 2048);
                const v4u hi = *(const v4u*)(smem + cur + (a_off ^ 64) + i * 2048);
                af[i] = (v8i){(int)lo[0], (int)lo[1], (int)lo[2], (int)lo[3], (int)hi[0], (int)hi[1], (int)hi[2], (int)hi[3]};
            }
#pragma unroll
            for (int i = 0; i < 2; ++i) {
                const v4u lo = *(const v4u*)(smem + cur + b_off + i * 2048);
                const v4u hi = *(const v4u*)(smem + cur + (b_off ^ 64) + i * 2048);
                bf[i] = (v8i){(int)lo[0], (int)lo[1], (int)lo[2], (int)lo[3], (int)hi[0], (int)hi[1], (int)hi[2], (int)hi[3]};
            }
#pragma unroll
            for (int i = 0; i < 4; ++i)
#pragma unroll
                for (int j = 0; j < 2; ++j)
                    acc[i][j] = __builtin_amdgcn_mfma_scale_f32_16x16x128_f8f6f4(af[i], bf[j], acc[i][j], 0, 0, 0,
                                                                                 0x7F7F7F7F, 0, 0x7F7F7F7F);
        }

        const bool bad = __hip_atomic_load(tmo, __ATOMIC_RELAXED, __HIP_MEMORY_SCOPE_AGENT) == call;
        const float s = bad ? __builtin_nanf("") : scale;
        if (owner == rank) {
            auto rs_o = __builtin_amdgcn_make_buffer_rsrc(P.out[rank], 0, (int)((size_t)MS * GN * 4), 0x00020000);
            const int voff = ((rnd * TM + wm * 32 + fr) * GN + rank * NS + nb * TN + half * GTN + wn * 64 + fq * 4) * 4;
#pragma unroll
            for (int j = 0; j < 2; ++j)
#pragma unroll
                for (int i = 0; i < 4; ++i) {
                    v4f v = acc[i][j] * s;
                    __builtin_amdgcn_raw_buffer_store_b128(__builtin_bit_cast(v4u, v), rs_o, voff + (j * 16 * GN + i * 16) * 4, 0, 16);
                }
        } else {
            const int tile = mb * NNB + nb;
            auto rs_y = __builtin_amdgcn_make_buffer_rsrc(myws + OFF_Y + (size_t)tile * (TM * TN * 4), 0, TM * TN * 4, 0x00020000);
            const int voff = ((wm * 32 + fr) * TN + half * GTN + wn * 64 + fq * 4) * 4;
#pragma unroll
            for (int j = 0; j < 2; ++j)
#pragma unroll
                for (int i = 0; i < 4; ++i) {
                    v4f v = acc[i][j] * s;
                    __builtin_amdgcn_raw_buffer_store_b128(__builtin_bit_cast(v4u, v), rs_y, voff + (j * 16 * TN + i * 16) * 4, 0, 17);
                }
            VMCNT0();
            __syncthreads();
            if (tid == 0)
                __hip_atomic_store((unsigned*)(P.ws[owner] + OFF_CTL) + C_YFLAG + ((rank * 32 + rnd * 8 + nb) * 2 + half) * CW,
                                   call, __ATOMIC_RELAXED, __HIP_MEMORY_SCOPE_SYSTEM);
        }
    }
}

}

extern "C" void kernel_launch(int rank, int nranks, ncclComm_t comm,
                              hipStream_t stream,
                              void* const* d_in, const int* in_sizes, int n_in,
                              void* d_out, int out_size,
                              const int* in_dtypes, int out_dtype,
                              void* const* d_out_all,
                              void* const* d_ws_all, size_t ws_bytes) {
    (void)comm; (void)in_sizes; (void)n_in; (void)in_dtypes; (void)out_dtype; (void)d_out;
    if (nranks != NR || (unsigned)rank >= (unsigned)NR || ws_bytes < WS_NEED || out_size != MS * GN) {
        fprintf(stderr, "unsupported configuration\n");
        return;
    }
    static bool init = false;
    if (!init) {
        HIP_CHECK(hipFuncSetAttribute((const void*)fused_kernel, hipFuncAttributeMaxDynamicSharedMemorySize, LDS_BYTES));
        init = true;
    }
    Params P;
    P.x = (const uint8_t*)d_in[0];
    P.w = (const uint8_t*)d_in[1];
    P.sx = (const float*)d_in[2];
    P.sw = (const float*)d_in[3];
    for (int p = 0; p < NR; ++p) {
        P.out[p] = (float*)d_out_all[p];
        P.ws[p] = (uint8_t*)d_ws_all[p];
    }
    P.rank = rank;
    fused_kernel<<<GRID, THREADS, LDS_BYTES, stream>>>(P);
    HIP_CHECK(hipGetLastError());
}
```

```cpp
#include <hip/hip_runtime.h>
#include <rccl/rccl.h>
#include <cstdio>
#include <cstdint>

#define HIP_CHECK(expr) do { hipError_t _e = (expr); if (_e != hipSuccess) { \
    fprintf(stderr, "HIP error %s at %s:%d\n", hipGetErrorString(_e), __FILE__, __LINE__); \
    return; } } while (0)

namespace {

typedef int v8i __attribute__((ext_vector_type(8)));
typedef float v4f __attribute__((ext_vector_type(4)));
typedef unsigned v4u __attribute__((ext_vector_type(4)));
typedef __attribute__((address_space(3))) volatile unsigned lds_u32;

constexpr int NR = 8;
constexpr int GM = 4096, GK = 4096, GN = 16384;
constexpr int KS = GK / NR;
constexpr int NS = GN / NR;
constexpr int MS = GM / NR;
constexpr int TM = 128, TN = 256, BK = 128;
constexpr int NMB = GM / TM;
constexpr int NNB = NS / TN;
constexpr int NTILES = NMB * NNB;
constexpr int GTN = 128;
constexpr int NGT = NTILES * 2;
constexpr int NKT = GK / BK;

constexpr int GRID = 256, THREADS = 512;
constexpr int NCOMM = 32;
constexpr int NPOOL = GRID - NCOMM;

constexpr size_t XS_BYTES = (size_t)GM * GK;
constexpr size_t WT_BYTES = (size_t)NS * GK;
constexpr size_t STAGE_BYTES = XS_BYTES + WT_BYTES;
constexpr size_t OFF_Y = 2 * STAGE_BYTES;
constexpr size_t Y_BYTES = (size_t)NTILES * TM * TN * 4;
constexpr size_t OFF_CTL = OFF_Y + Y_BYTES;
constexpr size_t WS_NEED = OFF_CTL + 262144;
constexpr int CW = 32;
constexpr int C_CALL = 0 * CW, C_TMO = 2 * CW, C_YCNT = 3 * CW;
constexpr int C_XCNT = 8 * CW;
constexpr int C_WCNT = 264 * CW;
constexpr int C_YFLAG = 328 * CW;
constexpr int C_TILEQ = 840 * CW;

constexpr int LW = NCOMM / 8;
constexpr int XU = 4;
constexpr int U_W0 = 8, U_X0 = U_W0 + 8 * XU, U_W1 = U_X0 + 56, U_XW = U_W1 + 24 * XU;
constexpr int NYU = 32 * 64;
constexpr unsigned XCNT_PER_CALL = XU;
constexpr unsigned WCNT_PER_CALL = 8;
constexpr int YW = 3;
constexpr unsigned YCNT_PER_CALL = (NR - 1) * NCOMM * YW;

constexpr int STAGE_LDS = (GTN + TM) * BK;
constexpr int NSTG = 4;
constexpr int LDS_CTL = NSTG * STAGE_LDS;
constexpr int LDS_BYTES = LDS_CTL + 64;

constexpr unsigned long long DEADLINE_TICKS = 50000000ull;

struct Params {
    const uint8_t* x;
    const uint8_t* w;
    const float* sx;
    const float* sw;
    float* out[NR];
    uint8_t* ws[NR];
    int rank;
};

template <bool SYS, int SLP = 2>
__device__ __forceinline__ void wait_ge(unsigned* p, unsigned target, unsigned* tmo, unsigned call) {
    unsigned long long t0 = __builtin_amdgcn_s_memrealtime();
    for (;;) {
        unsigned v = SYS ? __hip_atomic_load(p, __ATOMIC_RELAXED, __HIP_MEMORY_SCOPE_SYSTEM)
                         : __hip_atomic_load(p, __ATOMIC_RELAXED, __HIP_MEMORY_SCOPE_AGENT);
        if (v >= target) break;
        if (__hip_atomic_load(tmo, __ATOMIC_RELAXED, __HIP_MEMORY_SCOPE_AGENT) == call ||
            __builtin_amdgcn_s_memrealtime() - t0 > DEADLINE_TICKS) {
            __hip_atomic_store(tmo, call, __ATOMIC_RELAXED, __HIP_MEMORY_SCOPE_AGENT);
            break;
        }
        __builtin_amdgcn_s_sleep(SLP);
    }
}

__device__ __forceinline__ void tr4x4(unsigned r0, unsigned r1, unsigned r2, unsigned r3,
                                      unsigned& o0, unsigned& o1, unsigned& o2, unsigned& o3) {
    unsigned t0 = __builtin_amdgcn_perm(r1, r0, 0x05010400u);
    unsigned t1 = __builtin_amdgcn_perm(r1, r0, 0x07030602u);
    unsigned t2 = __builtin_amdgcn_perm(r3, r2, 0x05010400u);
    unsigned t3 = __builtin_amdgcn_perm(r3, r2, 0x07030602u);
    o0 = __builtin_amdgcn_perm(t2, t0, 0x05040100u);
    o1 = __builtin_amdgcn_perm(t2, t0, 0x07060302u);
    o2 = __builtin_amdgcn_perm(t3, t1, 0x05040100u);
    o3 = __builtin_amdgcn_perm(t3, t1, 0x07060302u);
}

#define VMCNT0() asm volatile("s_waitcnt vmcnt(0)" ::: "memory")
__device__ __forceinline__ void aload(v4u& d, const uint8_t* p) {
    asm volatile("global_load_dwordx4 %0, %1, off" : "=v"(d) : "v"(p) : "memory");
}
__device__ __forceinline__ void aload_sys(v4u& d, const uint8_t* p) {
    asm volatile("global_load_dwordx4 %0, %1, off sc0 sc1" : "=v"(d) : "v"(p) : "memory");
}
#define TIE4(a, i) "+v"(a[i]), "+v"(a[i + 1]), "+v"(a[i + 2]), "+v"(a[i + 3])
#define WAIT16(n, a) asm volatile("s_waitcnt vmcnt(" #n ")" : TIE4(a, 0), TIE4(a, 4), TIE4(a, 8), TIE4(a, 12) :: "memory")
#define WAIT14(n, a) asm volatile("s_waitcnt vmcnt(" #n ")" : TIE4(a, 0), TIE4(a, 4), TIE4(a, 8), "+v"(a[12]), "+v"(a[13]) :: "memory")

__device__ __forceinline__ int y_grab(lds_u32* sctl, int lane) {
    unsigned j = 0;
    if (lane == 0) j = __hip_atomic_fetch_add((__attribute__((address_space(3))) unsigned*)&sctl[4], 1u,
                                              __ATOMIC_RELAXED, __HIP_MEMORY_SCOPE_WORKGROUP);
    return __builtin_amdgcn_readfirstlane((int)j);
}
__device__ __forceinline__ bool y_ready(int wg, int j, lds_u32* sctl) {
    return (unsigned)__builtin_amdgcn_readfirstlane(sctl[2]) > (unsigned)((j * NCOMM + wg) >> 6);
}
__device__ __forceinline__ void y_load(v4u (&in)[14], const Params& P, int rank, int wg, int j, int lane) {
    const int yu = j * NCOMM + wg;
    const int k = yu >> 6, sub = yu & 63;
    const size_t off = OFF_Y + (size_t)(rank * 32 + k) * (TM * TN * 4) + sub * 2048 + lane * 16;
#pragma unroll
    for (int h = 0; h < 2; ++h)
#pragma unroll
        for (int q = 0; q < 7; ++q)
            aload_sys(in[h * 7 + q], P.ws[(rank + 1 + q) & 7] + off + h * 1024);
}
__device__ __forceinline__ void y_store(v4u (&in)[14], const Params& P, int rank, int wg, int j, int lane,
                                        lds_u32* sctl) {
    const int yu = j * NCOMM + wg;
    const int k = yu >> 6, sub = yu & 63;
    const int rnd = k >> 3, nb = k & 7;
    const bool bad = __builtin_amdgcn_readfirstlane(sctl[3]) != 0;
    const v4u nanv = {0x7fc00000u, 0x7fc00000u, 0x7fc00000u, 0x7fc00000u};
    auto rs = __builtin_amdgcn_make_buffer_rsrc(P.out[rank], 0, (int)((size_t)MS * GN * 4), 0x00020000);
    const int voff = ((rnd * TM + sub * 2) * GN + nb * TN) * 4 + lane * 16;
#pragma unroll
    for (int h = 0; h < 2; ++h)
#pragma unroll
        for (int q = 0; q < 7; ++q)
            __builtin_amdgcn_raw_buffer_store_b128(bad ? nanv : in[h * 7 + q], rs,
                                                   voff + h * (GN * 4) + ((rank + 1 + q) & 7) * (NS * 4), 0, 16);
}

__device__ __forceinline__ void comm_role(const Params& P, unsigned call, int wg, int wave, int lane,
                                          lds_u32* sctl) {
    const int rank = P.rank;
    uint8_t* myws = P.ws[rank];
    unsigned* myctl = (unsigned*)(myws + OFF_CTL);
    unsigned* tmo = myctl + C_TMO;
    const size_t par = (size_t)(call & 1u) * STAGE_BYTES;
    if (wave == 7) {
        for (int k = 0; k < 32; ++k) {
            if (lane < 14) wait_ge<true>(myctl + C_YFLAG + ((((rank + 1 + (lane >> 1)) & 7) * 32 + k) * 2 + (lane & 1)) * CW, call, tmo, call);
            __builtin_amdgcn_fence(__ATOMIC_ACQUIRE, "");
            const bool bad = __hip_atomic_load(tmo, __ATOMIC_RELAXED, __HIP_MEMORY_SCOPE_AGENT) == call;
            VMCNT0();
            if (lane == 0) { if (bad) sctl[3] = 1u; sctl[2] = (unsigned)k + 1u; }
        }
        if (wg == 0 && lane == 0) {
            wait_ge<true>(myctl + C_YCNT, call * YCNT_PER_CALL, tmo, call);
            if (__hip_atomic_load(tmo, __ATOMIC_RELAXED, __HIP_MEMORY_SCOPE_AGENT) == call)
                P.out[rank][(size_t)rank * NS] = __builtin_nanf("");
        }
        return;
    }
    const int JEND = (NYU - wg + NCOMM - 1) / NCOMM;
    v4u A[14], B[14];
    int j = 0;
    bool pre = false;
    {
        const int p = wg & 7;
        unsigned* dctl = (unsigned*)(P.ws[p] + OFF_CTL);
        auto rs = __builtin_amdgcn_make_buffer_rsrc(P.ws[p] + par, 0, (int)STAGE_BYTES, 0x00020000);
        for (int u = (wg >> 3) * 7 + wave; u < U_XW; u += 7 * LW) {
            unsigned* cnt;
            if (u < U_W0 || (u >= U_X0 && u < U_W1)) {
                const int wu = (u < U_W0) ? u : (u - U_X0 + 8);
                const int nb = wu >> 3, kb = (wu >> 1) & 3, nh = wu & 1;
                const int kg = lane & 7, ng = lane >> 3;
                const uint8_t* src = P.w + (size_t)(kb * 128 + kg * 16) * GN + (p * NS + nb * TN + nh * 128 + ng * 16);
                v4u in[16];
#pragma unroll
                for (int i = 0; i < 16; ++i) aload(in[i], src + (size_t)i * GN);
                WAIT16(0, in);
                v4u o[16];
#pragma unroll
                for (int a = 0; a < 4; ++a)
#pragma unroll
                    for (int b = 0; b < 4; ++b) {
                        unsigned q0, q1, q2, q3;
                        tr4x4(in[4 * a + 0][b], in[4 * a + 1][b], in[4 * a + 2][b], in[4 * a + 3][b], q0, q1, q2, q3);
                        o[4 * b + 0][a] = q0; o[4 * b + 1][a] = q1; o[4 * b + 2][a] = q2; o[4 * b + 3][a] = q3;
                    }
                const int voff = (int)XS_BYTES + rank * (NS * KS) + (nb * TN + nh * 128 + ng * 16) * KS + kb * 128 + kg * 16;
#pragma unroll
                for (int j = 0; j < 16; ++j)
                    __builtin_amdgcn_raw_buffer_store_b128(o[j], rs, voff + j * KS, 0, 17);
                cnt = dctl + C_WCNT + (nb * 8 + rank) * CW;
            } else {
                const int xu = (u < U_X0) ? (u - U_W0) : (u - U_W1 + 8 * XU);
                const int blk = xu / XU, sub = xu % XU;
                const int mb = ((p + 1 + (blk & 7)) & 7) * 4 + (blk >> 3);
                const int off = (mb * TM + sub * 32) * KS + lane * 16;
                v4u in[16];
#pragma unroll
                for (int i = 0; i < 16; ++i) in[i] = *(const v4u*)(P.x + off + i * 1024);
#pragma unroll
                for (int i = 0; i < 16; ++i)
                    __builtin_amdgcn_raw_buffer_store_b128(in[i], rs, rank * (GM * KS) + off + i * 1024, 0, 17);
                cnt = dctl + C_XCNT + (mb * 8 + rank) * CW;
            }
            if (u + 7 * LW >= U_XW && wave < YW) {
                j = y_grab(sctl, lane);
                pre = j < JEND && y_ready(wg, j, sctl);
                asm volatile("" ::: "memory");
                if (pre) y_load(A, P, rank, wg, j, lane);
                WAIT14(0, A);
            } else {
                VMCNT0();
            }
            if (lane == 0) __hip_atomic_fetch_add(cnt, 1u, __ATOMIC_RELAXED, __HIP_MEMORY_SCOPE_SYSTEM);
        }
    }
    if (wave < YW) {
        if (j < JEND) {
            if (!pre) {
                while (!y_ready(wg, j, sctl)) __builtin_amdgcn_s_sleep(1);
                asm volatile("" ::: "memory");
                y_load(A, P, rank, wg, j, lane);
                WAIT14(0, A);
            }
#define Y_HALF(CUR, NXT)                                                       \
            {                                                                  \
                const int j1 = y_grab(sctl, lane);                             \
                const bool more = j1 < JEND;                                   \
                const bool rdy = more && y_ready(wg, j1, sctl);                \
                asm volatile("" ::: "memory");                                 \
                if (rdy) y_load(NXT, P, rank, wg, j1, lane);                   \
                y_store(CUR, P, rank, wg, j, lane, sctl);                      \
                if (!more) break;                                              \
                if (rdy) { WAIT14(14, NXT); }                                  \
                else {                                                         \
                    while (!y_ready(wg, j1, sctl)) __builtin_amdgcn_s_sleep(1);\
                    asm volatile("" ::: "memory");                             \
                    y_load(NXT, P, rank, wg, j1, lane);                        \
                    WAIT14(0, NXT);                                            \
                }                                                              \
                j = j1;                                                        \
            }
            for (;;) {
                Y_HALF(A, B)
                Y_HALF(B, A)
            }
        }
        uint8_t* q = P.ws[0];
#pragma unroll
        for (int i = 1; i < NR; ++i) q = (((rank + 1 + lane) & 7) == i) ? P.ws[i] : q;
        if (lane < 7)
            __hip_atomic_fetch_add((unsigned*)(q + OFF_CTL) + C_YCNT, 1u, __ATOMIC_RELAXED, __HIP_MEMORY_SCOPE_SYSTEM);
    }
}

__global__ __launch_bounds__(THREADS, 2) void fused_kernel(Params P) {
    extern __shared__ __attribute__((aligned(16))) uint8_t smem[];
    const int tid = threadIdx.x;
    const int wave = __builtin_amdgcn_readfirstlane(tid >> 6);
    const int lane = tid & 63;
    const int rank = P.rank;
    uint8_t* myws = P.ws[rank];
    unsigned* myctl = (unsigned*)(myws + OFF_CTL);
    unsigned* tmo = myctl + C_TMO;
    lds_u32* sctl = (lds_u32*)(smem + LDS_CTL);

    if (tid == 0) {
        unsigned t = __hip_atomic_fetch_add(myctl + C_CALL, 1u, __ATOMIC_RELAXED, __HIP_MEMORY_SCOPE_AGENT);
        sctl[0] = t / GRID + 1u;
        sctl[2] = 0u;
        sctl[3] = 0u;
        sctl[4] = 0u;
    }
    __syncthreads();
    const unsigned call = __builtin_amdgcn_readfirstlane(sctl[0]);

    const int wg = blockIdx.x;
    if (wg < NCOMM) {
        comm_role(P, call, wg, wave, lane, sctl);
        return;
    }

    const float scale = P.sx[0] * P.sw[0];
    const uint8_t* stage = myws + (size_t)(call & 1u) * STAGE_BYTES;
    const int wn = wave >> 2, wm = wave & 3;
    const int fr = lane & 15, fq = lane >> 4;
    const int sw0 = ((fq) ^ (fr & 7)) * 16;
    const int a_off = (wn * 64 + fr) * BK + sw0;
    const int b_off = GTN * BK + (wm * 32 + fr) * BK + sw0;
    const int srow = lane >> 3;
    const int schunk = ((lane & 7) ^ (srow & 7)) * 16;

    for (;;) {
        __syncthreads();
        if (wave == 0) {
            unsigned t = 0;
            if (lane == 0) {
                t = __hip_atomic_fetch_add(myctl + C_TILEQ + (wg & 7) * CW, 1u, __ATOMIC_RELAXED, __HIP_MEMORY_SCOPE_AGENT);
                t -= (call - 1u) * (unsigned)((NGT + NPOOL) / 8);
                if (t < (unsigned)(NGT / 8)) {
                    const unsigned rnd_ = t >> 4, oi_ = (t >> 1) & 7, nb_ = wg & 7;
                    const unsigned t2_ = rnd_ * 64 + (rnd_ == 0 ? nb_ * 8 + oi_ : oi_ * 8 + nb_);
                    t = t2_ * 2 + (t & 1);
                } else {
                    t = NGT;
                }
                sctl[1] = t;
            }
            t = __builtin_amdgcn_readfirstlane(t);
            if (t < (unsigned)NGT) {
                const int t2 = t >> 1;
                const int nb = (t2 < 64) ? (t2 >> 3) : (t2 & 7), oi = (t2 < 64) ? (t2 & 7) : ((t2 >> 3) & 7);
                const int mb = ((rank + 1 + oi) & 7) * 4 + (t2 >> 6);
                if (lane < 16)
                    wait_ge<true, 16>(myctl + ((lane < 8) ? C_XCNT + (mb * 8 + lane) * CW : C_WCNT + (nb * 8 + lane - 8) * CW),
                                  call * ((lane < 8) ? XCNT_PER_CALL : WCNT_PER_CALL), tmo, call);
                __builtin_amdgcn_fence(__ATOMIC_ACQUIRE, "");
                VMCNT0();
            }
        }
        __syncthreads();
        const unsigned t = __builtin_amdgcn_readfirstlane(sctl[1]);
        if (t >= (unsigned)NGT) break;
        const int t2 = t >> 1, half = t & 1;
        const int rnd = t2 >> 6, nb = (t2 < 64) ? (t2 >> 3) : (t2 & 7), oi = (t2 < 64) ? (t2 & 7) : ((t2 >> 3) & 7);
        const int owner = (rank + 1 + oi) & 7;
        const int mb = owner * 4 + rnd;

        const uint8_t* gA = stage + XS_BYTES + (size_t)(nb * TN + half * GTN + wave * 16 + srow) * KS + schunk;
        const uint8_t* gB = stage + (size_t)(mb * TM + wave * 16 + srow) * KS + schunk;
        auto issue = [&](int kt, int buf) {
            uint8_t* la = smem + buf + wave * 2048;
            uint8_t* lb = smem + buf + GTN * BK + wave * 2048;
            const uint8_t* a = gA + (size_t)(kt >> 2) * (NS * KS) + (kt & 3) * BK;
            const uint8_t* b = gB + (size_t)(kt >> 2) * (GM * KS) + (kt & 3) * BK;
#pragma unroll
            for (int j = 0; j < 2; ++j)
                __builtin_amdgcn_global_load_lds((const unsigned*)(a + j * 8 * KS), (unsigned*)(la + j * 1024), 16, 0, 0);
#pragma unroll
            for (int j = 0; j < 2; ++j)
                __builtin_amdgcn_global_load_lds((const unsigned*)(b + j * 8 * KS), (unsigned*)(lb + j * 1024), 16, 0, 0);
        };

        v4f acc[4][2];
#pragma unroll
        for (int i = 0; i < 4; ++i)
#pragma unroll
            for (int j = 0; j < 2; ++j) acc[i][j] = (v4f){0.f, 0.f, 0.f, 0.f};

        issue(0, 0);
        issue(1, STAGE_LDS);
        issue(2, 2 * STAGE_LDS);
        for (int kt = 0; kt < NKT; ++kt) {
            if (kt < NKT - 2) asm volatile("s_waitcnt vmcnt(8)" ::: "memory");
            else if (kt == NKT - 2) asm volatile("s_waitcnt vmcnt(4)" ::: "memory");
            else asm volatile("s_waitcnt vmcnt(0)" ::: "memory");
            asm volatile("s_waitcnt lgkmcnt(0)" ::: "memory");
            __builtin_amdgcn_s_barrier();
            asm volatile("" ::: "memory");
            const int cur = (kt & 3) * STAGE_LDS;
            if (kt + 3 < NKT) issue(kt + 3, ((kt + 3) & 3) * STAGE_LDS);
            v8i af[4], bf[2];
#pragma unroll
            for (int i = 0; i < 4; ++i) {
                const v4u lo = *(const v4u*)(smem + cur + a_off + i * 2048);
                const v4u hi = *(const v4u*)(smem + cur + (a_off ^ 64) + i * 2048);
                af[i] = (v8i){(int)lo[0], (int)lo[1], (int)lo[2], (int)lo[3], (int)hi[0], (int)hi[1], (int)hi[2], (int)hi[3]};
            }
#pragma unroll
            for (int i = 0; i < 2; ++i) {
                const v4u lo = *(const v4u*)(smem + cur + b_off + i * 2048);
                const v4u hi = *(const v4u*)(smem + cur + (b_off ^ 64) + i * 2048);
                bf[i] = (v8i){(int)lo[0], (int)lo[1], (int)lo[2], (int)lo[3], (int)hi[0], (int)hi[1], (int)hi[2], (int)hi[3]};
            }
#pragma unroll
            for (int i = 0; i < 4; ++i)
#pragma unroll
                for (int j = 0; j < 2; ++j)
                    acc[i][j] = __builtin_amdgcn_mfma_scale_f32_16x16x128_f8f6f4(af[i], bf[j], acc[i][j], 0, 0, 0,
                                                                                 0x7F7F7F7F, 0, 0x7F7F7F7F);
        }

        const bool bad = __hip_atomic_load(tmo, __ATOMIC_RELAXED, __HIP_MEMORY_SCOPE_AGENT) == call;
        const float s = bad ? __builtin_nanf("") : scale;
        if (owner == rank) {
            auto rs_o = __builtin_amdgcn_make_buffer_rsrc(P.out[rank], 0, (int)((size_t)MS * GN * 4), 0x00020000);
            const int voff = ((rnd * TM + wm * 32 + fr) * GN + rank * NS + nb * TN + half * GTN + wn * 64 + fq * 4) * 4;
#pragma unroll
            for (int j = 0; j < 2; ++j)
#pragma unroll
                for (int i = 0; i < 4; ++i) {
                    v4f v = acc[i][j] * s;
                    __builtin_amdgcn_raw_buffer_store_b128(__builtin_bit_cast(v4u, v), rs_o, voff + (j * 16 * GN + i * 16) * 4, 0, 16);
                }
        } else {
            const int tile = mb * NNB + nb;
            auto rs_y = __builtin_amdgcn_make_buffer_rsrc(myws + OFF_Y + (size_t)tile * (TM * TN * 4), 0, TM * TN * 4, 0x00020000);
            const int voff = ((wm * 32 + fr) * TN + half * GTN + wn * 64 + fq * 4) * 4;
#pragma unroll
            for (int j = 0; j < 2; ++j)
#pragma unroll
                for (int i = 0; i < 4; ++i) {
                    v4f v = acc[i][j] * s;
                    __builtin_amdgcn_raw_buffer_store_b128(__builtin_bit_cast(v4u, v), rs_y, voff + (j * 16 * TN + i * 16) * 4, 0, 17);
                }
            VMCNT0();
            __syncthreads();
            if (tid == 0)
                __hip_atomic_store((unsigned*)(P.ws[owner] + OFF_CTL) + C_YFLAG + ((rank * 32 + rnd * 8 + nb) * 2 + half) * CW,
                                   call, __ATOMIC_RELAXED, __HIP_MEMORY_SCOPE_SYSTEM);
        }
    }
}

}

extern "C" void kernel_launch(int rank, int nranks, ncclComm_t comm,
                              hipStream_t stream,
                              void* const* d_in, const int* in_sizes, int n_in,
                              void* d_out, int out_size,
                              const int* in_dtypes, int out_dtype,
                              void* const* d_out_all,
                              void* const* d_ws_all, size_t ws_bytes) {
    (void)comm; (void)in_sizes; (void)n_in; (void)in_dtypes; (void)out_dtype; (void)d_out;
    if (nranks != NR || (unsigned)rank >= (unsigned)NR || ws_bytes < WS_NEED || out_size != MS * GN) {
        fprintf(stderr, "unsupported configuration\n");
        return;
    }
    static bool init = false;
    if (!init) {
        HIP_CHECK(hipFuncSetAttribute((const void*)fused_kernel, hipFuncAttributeMaxDynamicSharedMemorySize, LDS_BYTES));
        init = true;
    }
    Params P;
    P.x = (const uint8_t*)d_in[0];
    P.w = (const uint8_t*)d_in[1];
    P.sx = (const float*)d_in[2];
    P.sw = (const float*)d_in[3];
    for (int p = 0; p < NR; ++p) {
        P.out[p] = (float*)d_out_all[p];
        P.ws[p] = (uint8_t*)d_ws_all[p];
    }
    P.rank = rank;
    fused_kernel<<<GRID, THREADS, LDS_BYTES, stream>>>(P);
    HIP_CHECK(hipGetLastError());
}
```

```cpp
#include <hip/hip_runtime.h>
#include <rccl/rccl.h>
#include <cstdio>
#include <cstdint>

#define HIP_CHECK(expr) do { hipError_t _e = (expr); if (_e != hipSuccess) { \
    fprintf(stderr, "HIP error %s at %s:%d\n", hipGetErrorString(_e), __FILE__, __LINE__); \
    return; } } while (0)

namespace {

typedef int v8i __attribute__((ext_vector_type(8)));
typedef float v4f __attribute__((ext_vector_type(4)));
typedef unsigned v4u __attribute__((ext_vector_type(4)));
typedef __attribute__((address_space(3))) volatile unsigned lds_u32;

constexpr int NR = 8;
constexpr int GM = 4096, GK = 4096, GN = 16384;
constexpr int KS = GK / NR;
constexpr int NS = GN / NR;
constexpr int MS = GM / NR;
constexpr int TM = 128, TN = 256, BK = 128;
constexpr int NMB = GM / TM;
constexpr int NNB = NS / TN;
constexpr int NTILES = NMB * NNB;
constexpr int GTN = 128;
constexpr int NGT = NTILES * 2;
constexpr int NKT = GK / BK;

constexpr int GRID = 256, THREADS = 512;
constexpr int NCOMM = 32;
constexpr int NPOOL = GRID - NCOMM;

constexpr size_t XS_BYTES = (size_t)GM * GK;
constexpr size_t WT_BYTES = (size_t)NS * GK;
constexpr size_t STAGE_BYTES = XS_BYTES + WT_BYTES;
constexpr size_t OFF_Y = 2 * STAGE_BYTES;
constexpr size_t Y_BYTES = (size_t)NTILES * TM * TN * 4;
constexpr size_t OFF_CTL = OFF_Y + Y_BYTES;
constexpr size_t WS_NEED = OFF_CTL + 262144;
constexpr int CW = 32;
constexpr int C_CALL = 0 * CW, C_TMO = 2 * CW;
constexpr int C_XCNT = 8 * CW;
constexpr int C_WCNT = 264 * CW;
constexpr int C_YFLAG = 328 * CW;
constexpr int C_TILEQ = 840 * CW;

constexpr int LW = NCOMM / 8;
constexpr int XU = 4;
constexpr int U_W0 = 8, U_X0 = U_W0 + 8 * XU, U_W1 = U_X0 + 56, U_XW = U_W1 + 24 * XU;
constexpr int NYU = 32 * 64;
constexpr unsigned XCNT_PER_CALL = XU;
constexpr unsigned WCNT_PER_CALL = 8;
constexpr int YW = 4;

constexpr int STAGE_LDS = (GTN + TM) * BK;
constexpr int NSTG = 4;
constexpr int LDS_CTL = NSTG * STAGE_LDS;
constexpr int LDS_BYTES = LDS_CTL + 64;

constexpr unsigned long long DEADLINE_TICKS = 50000000ull;

struct Params {
    const uint8_t* x;
    const uint8_t* w;
    const float* sx;
    const float* sw;
    float* out[NR];
    uint8_t* ws[NR];
    int rank;
};

template <bool SYS, int SLP = 2>
__device__ __forceinline__ void wait_ge(unsigned* p, unsigned target, unsigned* tmo, unsigned call) {
    unsigned long long t0 = __builtin_amdgcn_s_memrealtime();
    for (;;) {
        unsigned v = SYS ? __hip_atomic_load(p, __ATOMIC_RELAXED, __HIP_MEMORY_SCOPE_SYSTEM)
                         : __hip_atomic_load(p, __ATOMIC_RELAXED, __HIP_MEMORY_SCOPE_AGENT);
        if (v >= target) break;
        if (__hip_atomic_load(tmo, __ATOMIC_RELAXED, __HIP_MEMORY_SCOPE_AGENT) == call ||
            __builtin_amdgcn_s_memrealtime() - t0 > DEADLINE_TICKS) {
            __hip_atomic_store(tmo, call, __ATOMIC_RELAXED, __HIP_MEMORY_SCOPE_AGENT);
            break;
        }
        __builtin_amdgcn_s_sleep(SLP);
    }
}

__device__ __forceinline__ void tr4x4(unsigned r0, unsigned r1, unsigned r2, unsigned r3,
                                      unsigned& o0, unsigned& o1, unsigned& o2, unsigned& o3) {
    unsigned t0 = __builtin_amdgcn_perm(r1, r0, 0x05010400u);
    unsigned t1 = __builtin_amdgcn_perm(r1, r0, 0x07030602u);
    unsigned t2 = __builtin_amdgcn_perm(r3, r2, 0x05010400u);
    unsigned t3 = __builtin_amdgcn_perm(r3, r2, 0x07030602u);
    o0 = __builtin_amdgcn_perm(t2, t0, 0x05040100u);
    o1 = __builtin_amdgcn_perm(t2, t0, 0x07060302u);
    o2 = __builtin_amdgcn_perm(t3, t1, 0x05040100u);
    o3 = __builtin_amdgcn_perm(t3, t1, 0x07060302u);
}

#define VMCNT0() asm volatile("s_waitcnt vmcnt(0)" ::: "memory")
__device__ __forceinline__ void aload(v4u& d, const uint8_t* p) {
    asm volatile("global_load_dwordx4 %0, %1, off" : "=v"(d) : "v"(p) : "memory");
}
__device__ __forceinline__ void aload_sys(v4u& d, const uint8_t* p) {
    asm volatile("global_load_dwordx4 %0, %1, off sc0 sc1" : "=v"(d) : "v"(p) : "memory");
}
#define TIE4(a, i) "+v"(a[i]), "+v"(a[i + 1]), "+v"(a[i + 2]), "+v"(a[i + 3])
#define WAIT16(n, a) asm volatile("s_waitcnt vmcnt(" #n ")" : TIE4(a, 0), TIE4(a, 4), TIE4(a, 8), TIE4(a, 12) :: "memory")
#define WAIT14(n, a) asm volatile("s_waitcnt vmcnt(" #n ")" : TIE4(a, 0), TIE4(a, 4), TIE4(a, 8), "+v"(a[12]), "+v"(a[13]) :: "memory")

__device__ __forceinline__ int y_grab(lds_u32* sctl, int lane) {
    unsigned j = 0;
    if (lane == 0) j = __hip_atomic_fetch_add((__attribute__((address_space(3))) unsigned*)&sctl[4], 1u,
                                              __ATOMIC_RELAXED, __HIP_MEMORY_SCOPE_WORKGROUP);
    return __builtin_amdgcn_readfirstlane((int)j);
}
__device__ __forceinline__ bool y_ready(int wg, int j, lds_u32* sctl) {
    return (unsigned)__builtin_amdgcn_readfirstlane(sctl[2]) > (unsigned)((j * NCOMM + wg) >> 6);
}
__device__ __forceinline__ void y_load(v4u (&in)[14], const Params& P, int rank, int wg, int j, int lane) {
    const int yu = j * NCOMM + wg;
    const int k = yu >> 6, sub = yu & 63;
    const size_t off = OFF_Y + (size_t)(rank * 32 + k) * (TM * TN * 4) + sub * 2048 + lane * 16;
#pragma unroll
    for (int h = 0; h < 2; ++h)
#pragma unroll
        for (int q = 0; q < 7; ++q)
            aload_sys(in[h * 7 + q], P.ws[(rank + 1 + q) & 7] + off + h * 1024);
}
__device__ __forceinline__ void y_store(v4u (&in)[14], const Params& P, int rank, int wg, int j, int lane,
                                        lds_u32* sctl) {
    const int yu = j * NCOMM + wg;
    const int k = yu >> 6, sub = yu & 63;
    const int rnd = k >> 3, nb = k & 7;
    const bool bad = __builtin_amdgcn_readfirstlane(sctl[3]) != 0;
    const v4u nanv = {0x7fc00000u, 0x7fc00000u, 0x7fc00000u, 0x7fc00000u};
    auto rs = __builtin_amdgcn_make_buffer_rsrc(P.out[rank], 0, (int)((size_t)MS * GN * 4), 0x00020000);
    const int voff = ((rnd * TM + sub * 2) * GN + nb * TN) * 4 + lane * 16;
#pragma unroll
    for (int h = 0; h < 2; ++h)
#pragma unroll
        for (int q = 0; q < 7; ++q)
            __builtin_amdgcn_raw_buffer_store_b128(bad ? nanv : in[h * 7 + q], rs,
                                                   voff + h * (GN * 4) + ((rank + 1 + q) & 7) * (NS * 4), 0, 16);
}

__device__ __forceinline__ void comm_role(const Params& P, unsigned call, int wg, int wave, int lane,
                                          lds_u32* sctl) {
    const int rank = P.rank;
    uint8_t* myws = P.ws[rank];
    unsigned* myctl = (unsigned*)(myws + OFF_CTL);
    unsigned* tmo = myctl + C_TMO;
    const size_t par = (size_t)(call & 1u) * STAGE_BYTES;
    if (wave == 7) {
        for (int k = 0; k < 32; ++k) {
            if (lane < 14) wait_ge<true>(myctl + C_YFLAG + ((((rank + 1 + (lane >> 1)) & 7) * 32 + k) * 2 + (lane & 1)) * CW, call, tmo, call);
            __builtin_amdgcn_fence(__ATOMIC_ACQUIRE, "");
            const bool bad = __hip_atomic_load(tmo, __ATOMIC_RELAXED, __HIP_MEMORY_SCOPE_AGENT) == call;
            VMCNT0();
            if (lane == 0) { if (bad) sctl[3] = 1u; sctl[2] = (unsigned)k + 1u; }
        }
        return;
    }
    const int JEND = (NYU - wg + NCOMM - 1) / NCOMM;
    v4u A[14], B[14];
    int j = 0;
    bool pre = false;
    {
        const int p = wg & 7;
        unsigned* dctl = (unsigned*)(P.ws[p] + OFF_CTL);
        auto rs = __builtin_amdgcn_make_buffer_rsrc(P.ws[p] + par, 0, (int)STAGE_BYTES, 0x00020000);
        for (int u = (wg >> 3) * 7 + wave; u < U_XW; u += 7 * LW) {
            unsigned* cnt;
            if (u < U_W0 || (u >= U_X0 && u < U_W1)) {
                const int wu = (u < U_W0) ? u : (u - U_X0 + 8);
                const int nb = wu >> 3, kb = (wu >> 1) & 3, nh = wu & 1;
                const int kg = lane & 7, ng = lane >> 3;
                const uint8_t* src = P.w + (size_t)(kb * 128 + kg * 16) * GN + (p * NS + nb * TN + nh * 128 + ng * 16);
                v4u in[16];
#pragma unroll
                for (int i = 0; i < 16; ++i) aload(in[i], src + (size_t)i * GN);
                WAIT16(0, in);
                v4u o[16];
#pragma unroll
                for (int a = 0; a < 4; ++a)
#pragma unroll
                    for (int b = 0; b < 4; ++b) {
                        unsigned q0, q1, q2, q3;
                        tr4x4(in[4 * a + 0][b], in[4 * a + 1][b], in[4 * a + 2][b], in[4 * a + 3][b], q0, q1, q2, q3);
                        o[4 * b + 0][a] = q0; o[4 * b + 1][a] = q1; o[4 * b + 2][a] = q2; o[4 * b + 3][a] = q3;
                    }
                const int voff = (int)XS_BYTES + rank * (NS * KS) + (nb * TN + nh * 128 + ng * 16) * KS + kb * 128 + kg * 16;
#pragma unroll
                for (int j = 0; j < 16; ++j)
                    __builtin_amdgcn_raw_buffer_store_b128(o[j], rs, voff + j * KS, 0, 17);
                cnt = dctl + C_WCNT + (nb * 8 + rank) * CW;
            } else {
                const int xu = (u < U_X0) ? (u - U_W0) : (u - U_W1 + 8 * XU);
                const int blk = xu / XU, sub = xu % XU;
                const int mb = ((p + 1 + (blk & 7)) & 7) * 4 + (blk >> 3);
                const int off = (mb * TM + sub * 32) * KS + lane * 16;
                v4u in[16];
#pragma unroll
                for (int i = 0; i < 16; ++i) in[i] = *(const v4u*)(P.x + off + i * 1024);
#pragma unroll
                for (int i = 0; i < 16; ++i)
                    __builtin_amdgcn_raw_buffer_store_b128(in[i], rs, rank * (GM * KS) + off + i * 1024, 0, 17);
                cnt = dctl + C_XCNT + (mb * 8 + rank) * CW;
            }
            if (u + 7 * LW >= U_XW && wave < YW) {
                j = y_grab(sctl, lane);
                pre = j < JEND && y_ready(wg, j, sctl);
                asm volatile("" ::: "memory");
                if (pre) y_load(A, P, rank, wg, j, lane);
                WAIT14(0, A);
            } else {
                VMCNT0();
            }
            if (lane == 0) __hip_atomic_fetch_add(cnt, 1u, __ATOMIC_RELAXED, __HIP_MEMORY_SCOPE_SYSTEM);
        }
    }
    if (wave < YW) {
        if (j < JEND) {
            if (!pre) {
                while (!y_ready(wg, j, sctl)) __builtin_amdgcn_s_sleep(1);
                asm volatile("" ::: "memory");
                y_load(A, P, rank, wg, j, lane);
                WAIT14(0, A);
            }
#define Y_HALF(CUR, NXT)                                                       \
            {                                                                  \
                const int j1 = y_grab(sctl, lane);                             \
                const bool more = j1 < JEND;                                   \
                const bool rdy = more && y_ready(wg, j1, sctl);                \
                asm volatile("" ::: "memory");                                 \
                if (rdy) y_load(NXT, P, rank, wg, j1, lane);                   \
                y_store(CUR, P, rank, wg, j, lane, sctl);                      \
                if (!more) break;                                              \
                if (rdy) { WAIT14(14, NXT); }                                  \
                else {                                                         \
                    while (!y_ready(wg, j1, sctl)) __builtin_amdgcn_s_sleep(1);\
                    asm volatile("" ::: "memory");                             \
                    y_load(NXT, P, rank, wg, j1, lane);                        \
                    WAIT14(0, NXT);                                            \
                }                                                              \
                j = j1;                                                        \
            }
            for (;;) {
                Y_HALF(A, B)
                Y_HALF(B, A)
            }
        }
    }
}

__global__ __launch_bounds__(THREADS, 2) void fused_kernel(Params P) {
    extern __shared__ __attribute__((aligned(16))) uint8_t smem[];
    const int tid = threadIdx.x;
    const int wave = __builtin_amdgcn_readfirstlane(tid >> 6);
    const int lane = tid & 63;
    const int rank = P.rank;
    uint8_t* myws = P.ws[rank];
    unsigned* myctl = (unsigned*)(myws + OFF_CTL);
    unsigned* tmo = myctl + C_TMO;
    lds_u32* sctl = (lds_u32*)(smem + LDS_CTL);

    if (tid == 0) {
        unsigned t = __hip_atomic_fetch_add(myctl + C_CALL, 1u, __ATOMIC_RELAXED, __HIP_MEMORY_SCOPE_AGENT);
        sctl[0] = t / GRID + 1u;
        sctl[2] = 0u;
        sctl[3] = 0u;
        sctl[4] = 0u;
    }
    __syncthreads();
    const unsigned call = __builtin_amdgcn_readfirstlane(sctl[0]);

    const int wg = blockIdx.x;
    if (wg < NCOMM) {
        comm_role(P, call, wg, wave, lane, sctl);
        return;
    }

    const float scale = P.sx[0] * P.sw[0];
    const uint8_t* stage = myws + (size_t)(call & 1u) * STAGE_BYTES;
    const int wn = wave >> 2, wm = wave & 3;
    const int fr = lane & 15, fq = lane >> 4;
    const int sw0 = ((fq) ^ (fr & 7)) * 16;
    const int a_off = (wn * 64 + fr) * BK + sw0;
    const int b_off = GTN * BK + (wm * 32 + fr) * BK + sw0;
    const int srow = lane >> 3;
    const int schunk = ((lane & 7) ^ (srow & 7)) * 16;

    for (;;) {
        __syncthreads();
        if (wave == 0) {
            unsigned t = 0;
            if (lane == 0) {
                t = __hip_atomic_fetch_add(myctl + C_TILEQ + (wg & 7) * CW, 1u, __ATOMIC_RELAXED, __HIP_MEMORY_SCOPE_AGENT);
                t -= (call - 1u) * (unsigned)((NGT + NPOOL) / 8);
                if (t < (unsigned)(NGT / 8)) {
                    const unsigned rnd_ = t >> 4, oi_ = (t >> 1) & 7, nb_ = wg & 7;
                    const unsigned t2_ = rnd_ * 64 + (rnd_ == 0 ? nb_ * 8 + oi_ : oi_ * 8 + nb_);
                    t = t2_ * 2 + (t & 1);
                } else {
                    t = NGT;
                }
                sctl[1] = t;
            }
            t = __builtin_amdgcn_readfirstlane(t);
            if (t < (unsigned)NGT) {
                const int t2 = t >> 1;
                const int nb = (t2 < 64) ? (t2 >> 3) : (t2 & 7), oi = (t2 < 64) ? (t2 & 7) : ((t2 >> 3) & 7);
                const int mb = ((rank + 1 + oi) & 7) * 4 + (t2 >> 6);
                if (lane < 16)
                    wait_ge<true, 16>(myctl + ((lane < 8) ? C_XCNT + (mb * 8 + lane) * CW : C_WCNT + (nb * 8 + lane - 8) * CW),
                                  call * ((lane < 8) ? XCNT_PER_CALL : WCNT_PER_CALL), tmo, call);
                __builtin_amdgcn_fence(__ATOMIC_ACQUIRE, "");
                VMCNT0();
            }
        }
        __syncthreads();
        const unsigned t = __builtin_amdgcn_readfirstlane(sctl[1]);
        if (t >= (unsigned)NGT) break;
        const int t2 = t >> 1, half = t & 1;
        const int rnd = t2 >> 6, nb = (t2 < 64) ? (t2 >> 3) : (t2 & 7), oi = (t2 < 64) ? (t2 & 7) : ((t2 >> 3) & 7);
        const int owner = (rank + 1 + oi) & 7;
        const int mb = owner * 4 + rnd;

        const uint8_t* gA = stage + XS_BYTES + (size_t)(nb * TN + half * GTN + wave * 16 + srow) * KS + schunk;
        const uint8_t* gB = stage + (size_t)(mb * TM + wave * 16 + srow) * KS + schunk;
        auto issue = [&](int kt, int buf) {
            uint8_t* la = smem + buf + wave * 2048;
            uint8_t* lb = smem + buf + GTN * BK + wave * 2048;
            const uint8_t* a = gA + (size_t)(kt >> 2) * (NS * KS) + (kt & 3) * BK;
            const uint8_t* b = gB + (size_t)(kt >> 2) * (GM * KS) + (kt & 3) * BK;
#pragma unroll
            for (int j = 0; j < 2; ++j)
                __builtin_amdgcn_global_load_lds((const unsigned*)(a + j * 8 * KS), (unsigned*)(la + j * 1024), 16, 0, 0);
#pragma unroll
            for (int j = 0; j < 2; ++j)
                __builtin_amdgcn_global_load_lds((const unsigned*)(b + j * 8 * KS), (unsigned*)(lb + j * 1024), 16, 0, 0);
        };

        v4f acc[4][2];
#pragma unroll
        for (int i = 0; i < 4; ++i)
#pragma unroll
            for (int j = 0; j < 2; ++j) acc[i][j] = (v4f){0.f, 0.f, 0.f, 0.f};

        issue(0, 0);
        issue(1, STAGE_LDS);
        issue(2, 2 * STAGE_LDS);
        for (int kt = 0; kt < NKT; ++kt) {
            if (kt < NKT - 2) asm volatile("s_waitcnt vmcnt(8)" ::: "memory");
            else if (kt == NKT - 2) asm volatile("s_waitcnt vmcnt(4)" ::: "memory");
            else asm volatile("s_waitcnt vmcnt(0)" ::: "memory");
            asm volatile("s_waitcnt lgkmcnt(0)" ::: "memory");
            __builtin_amdgcn_s_barrier();
            asm volatile("" ::: "memory");
            const int cur = (kt & 3) * STAGE_LDS;
            if (kt + 3 < NKT) issue(kt + 3, ((kt + 3) & 3) * STAGE_LDS);
            v8i af[4], bf[2];
#pragma unroll
            for (int i = 0; i < 4; ++i) {
                const v4u lo = *(const v4u*)(smem + cur + a_off + i * 2048);
                const v4u hi = *(const v4u*)(smem + cur + (a_off ^ 64) + i * 2048);
                af[i] = (v8i){(int)lo[0], (int)lo[1], (int)lo[2], (int)lo[3], (int)hi[0], (int)hi[1], (int)hi[2], (int)hi[3]};
            }
#pragma unroll
            for (int i = 0; i < 2; ++i) {
                const v4u lo = *(const v4u*)(smem + cur + b_off + i * 2048);
                const v4u hi = *(const v4u*)(smem + cur + (b_off ^ 64) + i * 2048);
                bf[i] = (v8i){(int)lo[0], (int)lo[1], (int)lo[2], (int)lo[3], (int)hi[0], (int)hi[1], (int)hi[2], (int)hi[3]};
            }
#pragma unroll
            for (int i = 0; i < 4; ++i)
#pragma unroll
                for (int j = 0; j < 2; ++j)
                    acc[i][j] = __builtin_amdgcn_mfma_scale_f32_16x16x128_f8f6f4(af[i], bf[j], acc[i][j], 0, 0, 0,
                                                                                 0x7F7F7F7F, 0, 0x7F7F7F7F);
        }

        const bool bad = __hip_atomic_load(tmo, __ATOMIC_RELAXED, __HIP_MEMORY_SCOPE_AGENT) == call;
        const float s = bad ? __builtin_nanf("") : scale;
        if (owner == rank) {
            auto rs_o = __builtin_amdgcn_make_buffer_rsrc(P.out[rank], 0, (int)((size_t)MS * GN * 4), 0x00020000);
            const int voff = ((rnd * TM + wm * 32 + fr) * GN + rank * NS + nb * TN + half * GTN + wn * 64 + fq * 4) * 4;
#pragma unroll
            for (int j = 0; j < 2; ++j)
#pragma unroll
                for (int i = 0; i < 4; ++i) {
                    v4f v = acc[i][j] * s;
                    __builtin_amdgcn_raw_buffer_store_b128(__builtin_bit_cast(v4u, v), rs_o, voff + (j * 16 * GN + i * 16) * 4, 0, 16);
                }
        } else {
            const int tile = mb * NNB + nb;
            auto rs_y = __builtin_amdgcn_make_buffer_rsrc(myws + OFF_Y + (size_t)tile * (TM * TN * 4), 0, TM * TN * 4, 0x00020000);
            const int voff = ((wm * 32 + fr) * TN + half * GTN + wn * 64 + fq * 4) * 4;
#pragma unroll
            for (int j = 0; j < 2; ++j)
#pragma unroll
                for (int i = 0; i < 4; ++i) {
                    v4f v = acc[i][j] * s;
                    __builtin_amdgcn_raw_buffer_store_b128(__builtin_bit_cast(v4u, v), rs_y, voff + (j * 16 * TN + i * 16) * 4, 0, 17);
                }
            VMCNT0();
            __syncthreads();
            if (tid == 0)
                __hip_atomic_store((unsigned*)(P.ws[owner] + OFF_CTL) + C_YFLAG + ((rank * 32 + rnd * 8 + nb) * 2 + half) * CW,
                                   call, __ATOMIC_RELAXED, __HIP_MEMORY_SCOPE_SYSTEM);
        }
    }
}

}

extern "C" void kernel_launch(int rank, int nranks, ncclComm_t comm,
                              hipStream_t stream,
                              void* const* d_in, const int* in_sizes, int n_in,
                              void* d_out, int out_size,
                              const int* in_dtypes, int out_dtype,
                              void* const* d_out_all,
                              void* const* d_ws_all, size_t ws_bytes) {
    (void)comm; (void)in_sizes; (void)n_in; (void)in_dtypes; (void)out_dtype; (void)d_out;
    if (nranks != NR || (unsigned)rank >= (unsigned)NR || ws_bytes < WS_NEED || out_size != MS * GN) {
        fprintf(stderr, "unsupported configuration\n");
        return;
    }
    static bool init = false;
    if (!init) {
        HIP_CHECK(hipFuncSetAttribute((const void*)fused_kernel, hipFuncAttributeMaxDynamicSharedMemorySize, LDS_BYTES));
        init = true;
    }
    Params P;
    P.x = (const uint8_t*)d_in[0];
    P.w = (const uint8_t*)d_in[1];
    P.sx = (const float*)d_in[2];
    P.sw = (const float*)d_in[3];
    for (int p = 0; p < NR; ++p) {
        P.out[p] = (float*)d_out_all[p];
        P.ws[p] = (uint8_t*)d_ws_all[p];
    }
    P.rank = rank;
    fused_kernel<<<GRID, THREADS, LDS_BYTES, stream>>>(P);
    HIP_CHECK(hipGetLastError());
}
```

```cpp
#include <hip/hip_runtime.h>
#include <rccl/rccl.h>
#include <cstdio>
#include <cstdint>

#define HIP_CHECK(expr) do { hipError_t _e = (expr); if (_e != hipSuccess) { \
    fprintf(stderr, "HIP error %s at %s:%d\n", hipGetErrorString(_e), __FILE__, __LINE__); \
    return; } } while (0)

namespace {

typedef int v8i __attribute__((ext_vector_type(8)));
typedef float v4f __attribute__((ext_vector_type(4)));
typedef unsigned v4u __attribute__((ext_vector_type(4)));
typedef __attribute__((address_space(3))) volatile unsigned lds_u32;

constexpr int NR = 8;
constexpr int GM = 4096, GK = 4096, GN = 16384;
constexpr int KS = GK / NR;
constexpr int NS = GN / NR;
constexpr int MS = GM / NR;
constexpr int TM = 128, TN = 256, BK = 128;
constexpr int NMB = GM / TM;
constexpr int NNB = NS / TN;
constexpr int NTILES = NMB * NNB;
constexpr int GTN = 128;
constexpr int NGT = NTILES * 2;
constexpr int NKT = GK / BK;

constexpr int GRID = 256, THREADS = 512;
constexpr int NCOMM = 32;
constexpr int NPOOL = GRID - NCOMM;

constexpr size_t XS_BYTES = (size_t)GM * GK;
constexpr size_t WT_BYTES = (size_t)NS * GK;
constexpr size_t STAGE_BYTES = XS_BYTES + WT_BYTES;
constexpr size_t OFF_Y = 2 * STAGE_BYTES;
constexpr size_t Y_BYTES = (size_t)NTILES * TM * TN * 4;
constexpr size_t OFF_CTL = OFF_Y + Y_BYTES;
constexpr size_t WS_NEED = OFF_CTL + 262144;
constexpr int CW = 32;
constexpr int C_CALL = 0 * CW, C_TMO = 2 * CW;
constexpr int C_XCNT = 8 * CW;
constexpr int C_WCNT = 264 * CW;
constexpr int C_YFLAG = 328 * CW;
constexpr int C_TILEQ = 840 * CW;

constexpr int LW = NCOMM / 8;
constexpr int XU = 4;
constexpr int U_W0 = 8, U_X0 = U_W0 + 8 * XU, U_W1 = U_X0 + 56, U_XW = U_W1 + 24 * XU;
#define XW_U1 140
constexpr int U1 = XW_U1;
constexpr int NYU = 32 * 64;
constexpr unsigned XCNT_PER_CALL = XU;
constexpr unsigned WCNT_PER_CALL = 8;
constexpr int YW = 4;

constexpr int STAGE_LDS = (GTN + TM) * BK;
constexpr int NSTG = 4;
constexpr int LDS_CTL = NSTG * STAGE_LDS;
constexpr int LDS_BYTES = LDS_CTL + 64;

constexpr unsigned long long DEADLINE_TICKS = 50000000ull;

struct Params {
    const uint8_t* x;
    const uint8_t* w;
    const float* sx;
    const float* sw;
    float* out[NR];
    uint8_t* ws[NR];
    int rank;
};

template <bool SYS, int SLP = 2>
__device__ __forceinline__ void wait_ge(unsigned* p, unsigned target, unsigned* tmo, unsigned call) {
    unsigned long long t0 = __builtin_amdgcn_s_memrealtime();
    for (;;) {
        unsigned v = SYS ? __hip_atomic_load(p, __ATOMIC_RELAXED, __HIP_MEMORY_SCOPE_SYSTEM)
                         : __hip_atomic_load(p, __ATOMIC_RELAXED, __HIP_MEMORY_SCOPE_AGENT);
        if (v >= target) break;
        if (__hip_atomic_load(tmo, __ATOMIC_RELAXED, __HIP_MEMORY_SCOPE_AGENT) == call ||
            __builtin_amdgcn_s_memrealtime() - t0 > DEADLINE_TICKS) {
            __hip_atomic_store(tmo, call, __ATOMIC_RELAXED, __HIP_MEMORY_SCOPE_AGENT);
            break;
        }
        __builtin_amdgcn_s_sleep(SLP);
    }
}

__device__ __forceinline__ void tr4x4(unsigned r0, unsigned r1, unsigned r2, unsigned r3,
                                      unsigned& o0, unsigned& o1, unsigned& o2, unsigned& o3) {
    unsigned t0 = __builtin_amdgcn_perm(r1, r0, 0x05010400u);
    unsigned t1 = __builtin_amdgcn_perm(r1, r0, 0x07030602u);
    unsigned t2 = __builtin_amdgcn_perm(r3, r2, 0x05010400u);
    unsigned t3 = __builtin_amdgcn_perm(r3, r2, 0x07030602u);
    o0 = __builtin_amdgcn_perm(t2, t0, 0x05040100u);
    o1 = __builtin_amdgcn_perm(t2, t0, 0x07060302u);
    o2 = __builtin_amdgcn_perm(t3, t1, 0x05040100u);
    o3 = __builtin_amdgcn_perm(t3, t1, 0x07060302u);
}

#define VMCNT0() asm volatile("s_waitcnt vmcnt(0)" ::: "memory")
__device__ __forceinline__ void aload(v4u& d, const uint8_t* p) {
    asm volatile("global_load_dwordx4 %0, %1, off" : "=v"(d) : "v"(p) : "memory");
}
__device__ __forceinline__ void aload_sys(v4u& d, const uint8_t* p) {
    asm volatile("global_load_dwordx4 %0, %1, off sc0 sc1" : "=v"(d) : "v"(p) : "memory");
}
#define TIE4(a, i) "+v"(a[i]), "+v"(a[i + 1]), "+v"(a[i + 2]), "+v"(a[i + 3])
#define WAIT16(n, a) asm volatile("s_waitcnt vmcnt(" #n ")" : TIE4(a, 0), TIE4(a, 4), TIE4(a, 8), TIE4(a, 12) :: "memory")
#define WAIT14(n, a) asm volatile("s_waitcnt vmcnt(" #n ")" : TIE4(a, 0), TIE4(a, 4), TIE4(a, 8), "+v"(a[12]), "+v"(a[13]) :: "memory")

__device__ __forceinline__ int y_grab(lds_u32* sctl, int lane) {
    unsigned j = 0;
    if (lane == 0) j = __hip_atomic_fetch_add((__attribute__((address_space(3))) unsigned*)&sctl[4], 1u,
                                              __ATOMIC_RELAXED, __HIP_MEMORY_SCOPE_WORKGROUP);
    return __builtin_amdgcn_readfirstlane((int)j);
}
__device__ __forceinline__ bool y_ready(int wg, int j, lds_u32* sctl) {
    return (unsigned)__builtin_amdgcn_readfirstlane(sctl[2]) > (unsigned)((j * NCOMM + wg) >> 6);
}
__device__ __forceinline__ void y_load(v4u (&in)[14], const Params& P, int rank, int wg, int j, int lane) {
    const int yu = j * NCOMM + wg;
    const int k = yu >> 6, sub = yu & 63;
    const size_t off = OFF_Y + (size_t)(rank * 32 + k) * (TM * TN * 4) + sub * 2048 + lane * 16;
#pragma unroll
    for (int h = 0; h < 2; ++h)
#pragma unroll
        for (int q = 0; q < 7; ++q)
            aload_sys(in[h * 7 + q], P.ws[(rank + 1 + q) & 7] + off + h * 1024);
}
__device__ __forceinline__ void y_store(v4u (&in)[14], const Params& P, int rank, int wg, int j, int lane,
                                        lds_u32* sctl) {
    const int yu = j * NCOMM + wg;
    const int k = yu >> 6, sub = yu & 63;
    const int rnd = k >> 3, nb = k & 7;
    const bool bad = __builtin_amdgcn_readfirstlane(sctl[3]) != 0;
    const v4u nanv = {0x7fc00000u, 0x7fc00000u, 0x7fc00000u, 0x7fc00000u};
    auto rs = __builtin_amdgcn_make_buffer_rsrc(P.out[rank], 0, (int)((size_t)MS * GN * 4), 0x00020000);
    const int voff = ((rnd * TM + sub * 2) * GN + nb * TN) * 4 + lane * 16;
#pragma unroll
    for (int h = 0; h < 2; ++h)
#pragma unroll
        for (int q = 0; q < 7; ++q)
            __builtin_amdgcn_raw_buffer_store_b128(bad ? nanv : in[h * 7 + q], rs,
                                                   voff + h * (GN * 4) + ((rank + 1 + q) & 7) * (NS * 4), 0, 16);
}

__device__ __forceinline__ void comm_role(const Params& P, unsigned call, int wg, int wave, int lane,
                                          lds_u32* sctl) {
    const int rank = P.rank;
    uint8_t* myws = P.ws[rank];
    unsigned* myctl = (unsigned*)(myws + OFF_CTL);
    unsigned* tmo = myctl + C_TMO;
    const size_t par = (size_t)(call & 1u) * STAGE_BYTES;
    if (wave == 7) {
        for (int k = 0; k < 32; ++k) {
            if (lane < 14) wait_ge<true>(myctl + C_YFLAG + ((((rank + 1 + (lane >> 1)) & 7) * 32 + k) * 2 + (lane & 1)) * CW, call, tmo, call);
            __builtin_amdgcn_fence(__ATOMIC_ACQUIRE, "");
            const bool bad = __hip_atomic_load(tmo, __ATOMIC_RELAXED, __HIP_MEMORY_SCOPE_AGENT) == call;
            VMCNT0();
            if (lane == 0) { if (bad) sctl[3] = 1u; sctl[2] = (unsigned)k + 1u; }
        }
        return;
    }
    const int JEND = (NYU - wg + NCOMM - 1) / NCOMM;
    v4u A[14], B[14];
    int j = 0;
    bool pre = false;
    {
        const int p = wg & 7;
        unsigned* dctl = (unsigned*)(P.ws[p] + OFF_CTL);
        auto rs = __builtin_amdgcn_make_buffer_rsrc(P.ws[p] + par, 0, (int)STAGE_BYTES, 0x00020000);
        int u = (wg >> 3) * 7 + wave;
        while (u < U_XW) {
            unsigned* cnt;
            if (u < U_W0 || (u >= U_X0 && u < U_W1)) {
                const int wu = (u < U_W0) ? u : (u - U_X0 + 8);
                const int nb = wu >> 3, kb = (wu >> 1) & 3, nh = wu & 1;
                const int kg = lane & 7, ng = lane >> 3;
                const uint8_t* src = P.w + (size_t)(kb * 128 + kg * 16) * GN + (p * NS + nb * TN + nh * 128 + ng * 16);
                v4u in[16];
#pragma unroll
                for (int i = 0; i < 16; ++i) aload(in[i], src + (size_t)i * GN);
                WAIT16(0, in);
                v4u o[16];
#pragma unroll
                for (int a = 0; a < 4; ++a)
#pragma unroll
                    for (int b = 0; b < 4; ++b) {
                        unsigned q0, q1, q2, q3;
                        tr4x4(in[4 * a + 0][b], in[4 * a + 1][b], in[4 * a + 2][b], in[4 * a + 3][b], q0, q1, q2, q3);
                        o[4 * b + 0][a] = q0; o[4 * b + 1][a] = q1; o[4 * b + 2][a] = q2; o[4 * b + 3][a] = q3;
                    }
                const int voff = (int)XS_BYTES + rank * (NS * KS) + (nb * TN + nh * 128 + ng * 16) * KS + kb * 128 + kg * 16;
#pragma unroll
                for (int j = 0; j < 16; ++j)
                    __builtin_amdgcn_raw_buffer_store_b128(o[j], rs, voff + j * KS, 0, 17);
                cnt = dctl + C_WCNT + (nb * 8 + rank) * CW;
            } else {
                const int xu = (u < U_X0) ? (u - U_W0) : (u - U_W1 + 8 * XU);
                const int blk = xu / XU, sub = xu % XU;
                const int mb = ((p + 1 + (blk & 7)) & 7) * 4 + (blk >> 3);
                const int off = (mb * TM + sub * 32) * KS + lane * 16;
                v4u in[16];
#pragma unroll
                for (int i = 0; i < 16; ++i) in[i] = *(const v4u*)(P.x + off + i * 1024);
#pragma unroll
                for (int i = 0; i < 16; ++i)
                    __builtin_amdgcn_raw_buffer_store_b128(in[i], rs, rank * (GM * KS) + off + i * 1024, 0, 17);
                cnt = dctl + C_XCNT + (mb * 8 + rank) * CW;
            }
            if (u + 7 * LW >= U1 && wave < YW) {
                j = y_grab(sctl, lane);
                pre = j < JEND && y_ready(wg, j, sctl);
                asm volatile("" ::: "memory");
                if (pre) y_load(A, P, rank, wg, j, lane);
                WAIT14(0, A);
            } else {
                VMCNT0();
            }
            if (lane == 0) __hip_atomic_fetch_add(cnt, 1u, __ATOMIC_RELAXED, __HIP_MEMORY_SCOPE_SYSTEM);
            if (u >= U1) u += (7 - YW) * LW;
            else if (u + 7 * LW < U1) u += 7 * LW;
            else u = (wave < YW) ? U_XW : U1 + (wg >> 3) * (7 - YW) + (wave - YW);
        }
    }
    if (wave < YW) {
        if (j < JEND) {
            if (!pre) {
                while (!y_ready(wg, j, sctl)) __builtin_amdgcn_s_sleep(1);
                asm volatile("" ::: "memory");
                y_load(A, P, rank, wg, j, lane);
                WAIT14(0, A);
            }
#define Y_HALF(CUR, NXT)                                                       \
            {                                                                  \
                const int j1 = y_grab(sctl, lane);                             \
                const bool more = j1 < JEND;                                   \
                const bool rdy = more && y_ready(wg, j1, sctl);                \
                asm volatile("" ::: "memory");                                 \
                if (rdy) y_load(NXT, P, rank, wg, j1, lane);                   \
                y_store(CUR, P, rank, wg, j, lane, sctl);                      \
                if (!more) break;                                              \
                if (rdy) { WAIT14(14, NXT); }                                  \
                else {                                                         \
                    while (!y_ready(wg, j1, sctl)) __builtin_amdgcn_s_sleep(1);\
                    asm volatile("" ::: "memory");                             \
                    y_load(NXT, P, rank, wg, j1, lane);                        \
                    WAIT14(0, NXT);                                            \
                }                                                              \
                j = j1;                                                        \
            }
            for (;;) {
                Y_HALF(A, B)
                Y_HALF(B, A)
            }
        }
    }
}

__global__ __launch_bounds__(THREADS, 2) void fused_kernel(Params P) {
    extern __shared__ __attribute__((aligned(16))) uint8_t smem[];
    const int tid = threadIdx.x;
    const int wave = __builtin_amdgcn_readfirstlane(tid >> 6);
    const int lane = tid & 63;
    const int rank = P.rank;
    uint8_t* myws = P.ws[rank];
    unsigned* myctl = (unsigned*)(myws + OFF_CTL);
    unsigned* tmo = myctl + C_TMO;
    lds_u32* sctl = (lds_u32*)(smem + LDS_CTL);

    if (tid == 0) {
        unsigned t = __hip_atomic_fetch_add(myctl + C_CALL, 1u, __ATOMIC_RELAXED, __HIP_MEMORY_SCOPE_AGENT);
        sctl[0] = t / GRID + 1u;
        sctl[2] = 0u;
        sctl[3] = 0u;
        sctl[4] = 0u;
    }
    __syncthreads();
    const unsigned call = __builtin_amdgcn_readfirstlane(sctl[0]);

    const int wg = blockIdx.x;
    if (wg < NCOMM) {
        comm_role(P, call, wg, wave, lane, sctl);
        return;
    }

    const float scale = P.sx[0] * P.sw[0];
    const uint8_t* stage = myws + (size_t)(call & 1u) * STAGE_BYTES;
    const int wn = wave >> 2, wm = wave & 3;
    const int fr = lane & 15, fq = lane >> 4;
    const int sw0 = ((fq) ^ (fr & 7)) * 16;
    const int a_off = (wn * 64 + fr) * BK + sw0;
    const int b_off = GTN * BK + (wm * 32 + fr) * BK + sw0;
    const int srow = lane >> 3;
    const int schunk = ((lane & 7) ^ (srow & 7)) * 16;

    for (;;) {
        __syncthreads();
        if (wave == 0) {
            unsigned t = 0;
            if (lane == 0) {
                t = __hip_atomic_fetch_add(myctl + C_TILEQ + (wg & 7) * CW, 1u, __ATOMIC_RELAXED, __HIP_MEMORY_SCOPE_AGENT);
                t -= (call - 1u) * (unsigned)((NGT + NPOOL) / 8);
                if (t < (unsigned)(NGT / 8)) {
                    const unsigned rnd_ = t >> 4, oi_ = (t >> 1) & 7, nb_ = wg & 7;
                    const unsigned t2_ = rnd_ * 64 + (rnd_ == 0 ? nb_ * 8 + oi_ : oi_ * 8 + nb_);
                    t = t2_ * 2 + (t & 1);
                } else {
                    t = NGT;
                }
                sctl[1] = t;
            }
            t = __builtin_amdgcn_readfirstlane(t);
            if (t < (unsigned)NGT) {
                const int t2 = t >> 1;
                const int nb = (t2 < 64) ? (t2 >> 3) : (t2 & 7), oi = (t2 < 64) ? (t2 & 7) : ((t2 >> 3) & 7);
                const int mb = ((rank + 1 + oi) & 7) * 4 + (t2 >> 6);
                if (lane < 16)
                    wait_ge<true, 16>(myctl + ((lane < 8) ? C_XCNT + (mb * 8 + lane) * CW : C_WCNT + (nb * 8 + lane - 8) * CW),
                                  call * ((lane < 8) ? XCNT_PER_CALL : WCNT_PER_CALL), tmo, call);
                __builtin_amdgcn_fence(__ATOMIC_ACQUIRE, "");
                VMCNT0();
            }
        }
        __syncthreads();
        const unsigned t = __builtin_amdgcn_readfirstlane(sctl[1]);
        if (t >= (unsigned)NGT) break;
        const int t2 = t >> 1, half = t & 1;
        const int rnd = t2 >> 6, nb = (t2 < 64) ? (t2 >> 3) : (t2 & 7), oi = (t2 < 64) ? (t2 & 7) : ((t2 >> 3) & 7);
        const int owner = (rank + 1 + oi) & 7;
        const int mb = owner * 4 + rnd;

        const uint8_t* gA = stage + XS_BYTES + (size_t)(nb * TN + half * GTN + wave * 16 + srow) * KS + schunk;
        const uint8_t* gB = stage + (size_t)(mb * TM + wave * 16 + srow) * KS + schunk;
        auto issue = [&](int kt, int buf) {
            uint8_t* la = smem + buf + wave * 2048;
            uint8_t* lb = smem + buf + GTN * BK + wave * 2048;
            const uint8_t* a = gA + (size_t)(kt >> 2) * (NS * KS) + (kt & 3) * BK;
            const uint8_t* b = gB + (size_t)(kt >> 2) * (GM * KS) + (kt & 3) * BK;
#pragma unroll
            for (int j = 0; j < 2; ++j)
                __builtin_amdgcn_global_load_lds((const unsigned*)(a + j * 8 * KS), (unsigned*)(la + j * 1024), 16, 0, 0);
#pragma unroll
            for (int j = 0; j < 2; ++j)
                __builtin_amdgcn_global_load_lds((const unsigned*)(b + j * 8 * KS), (unsigned*)(lb + j * 1024), 16, 0, 0);
        };

        v4f acc[4][2];
#pragma unroll
        for (int i = 0; i < 4; ++i)
#pragma unroll
            for (int j = 0; j < 2; ++j) acc[i][j] = (v4f){0.f, 0.f, 0.f, 0.f};

        issue(0, 0);
        issue(1, STAGE_LDS);
        issue(2, 2 * STAGE_LDS);
        for (int kt = 0; kt < NKT; ++kt) {
            if (kt < NKT - 2) asm volatile("s_waitcnt vmcnt(8)" ::: "memory");
            else if (kt == NKT - 2) asm volatile("s_waitcnt vmcnt(4)" ::: "memory");
            else asm volatile("s_waitcnt vmcnt(0)" ::: "memory");
            asm volatile("s_waitcnt lgkmcnt(0)" ::: "memory");
            __builtin_amdgcn_s_barrier();
            asm volatile("" ::: "memory");
            const int cur = (kt & 3) * STAGE_LDS;
            if (kt + 3 < NKT) issue(kt + 3, ((kt + 3) & 3) * STAGE_LDS);
            v8i af[4], bf[2];
#pragma unroll
            for (int i = 0; i < 4; ++i) {
                const v4u lo = *(const v4u*)(smem + cur + a_off + i * 2048);
                const v4u hi = *(const v4u*)(smem + cur + (a_off ^ 64) + i * 2048);
                af[i] = (v8i){(int)lo[0], (int)lo[1], (int)lo[2], (int)lo[3], (int)hi[0], (int)hi[1], (int)hi[2], (int)hi[3]};
            }
#pragma unroll
            for (int i = 0; i < 2; ++i) {
                const v4u lo = *(const v4u*)(smem + cur + b_off + i * 2048);
                const v4u hi = *(const v4u*)(smem + cur + (b_off ^ 64) + i * 2048);
                bf[i] = (v8i){(int)lo[0], (int)lo[1], (int)lo[2], (int)lo[3], (int)hi[0], (int)hi[1], (int)hi[2], (int)hi[3]};
            }
#pragma unroll
            for (int i = 0; i < 4; ++i)
#pragma unroll
                for (int j = 0; j < 2; ++j)
                    acc[i][j] = __builtin_amdgcn_mfma_scale_f32_16x16x128_f8f6f4(af[i], bf[j], acc[i][j], 0, 0, 0,
                                                                                 0x7F7F7F7F, 0, 0x7F7F7F7F);
        }

        const bool bad = __hip_atomic_load(tmo, __ATOMIC_RELAXED, __HIP_MEMORY_SCOPE_AGENT) == call;
        const float s = bad ? __builtin_nanf("") : scale;
        if (owner == rank) {
            auto rs_o = __builtin_amdgcn_make_buffer_rsrc(P.out[rank], 0, (int)((size_t)MS * GN * 4), 0x00020000);
            const int voff = ((rnd * TM + wm * 32 + fr) * GN + rank * NS + nb * TN + half * GTN + wn * 64 + fq * 4) * 4;
#pragma unroll
            for (int j = 0; j < 2; ++j)
#pragma unroll
                for (int i = 0; i < 4; ++i) {
                    v4f v = acc[i][j] * s;
                    __builtin_amdgcn_raw_buffer_store_b128(__builtin_bit_cast(v4u, v), rs_o, voff + (j * 16 * GN + i * 16) * 4, 0, 16);
                }
        } else {
            const int tile = mb * NNB + nb;
            auto rs_y = __builtin_amdgcn_make_buffer_rsrc(myws + OFF_Y + (size_t)tile * (TM * TN * 4), 0, TM * TN * 4, 0x00020000);
            const int voff = ((wm * 32 + fr) * TN + half * GTN + wn * 64 + fq * 4) * 4;
#pragma unroll
            for (int j = 0; j < 2; ++j)
#pragma unroll
                for (int i = 0; i < 4; ++i) {
                    v4f v = acc[i][j] * s;
                    __builtin_amdgcn_raw_buffer_store_b128(__builtin_bit_cast(v4u, v), rs_y, voff + (j * 16 * TN + i * 16) * 4, 0, 17);
                }
            VMCNT0();
            __syncthreads();
            if (tid == 0)
                __hip_atomic_store((unsigned*)(P.ws[owner] + OFF_CTL) + C_YFLAG + ((rank * 32 + rnd * 8 + nb) * 2 + half) * CW,
                                   call, __ATOMIC_RELAXED, __HIP_MEMORY_SCOPE_SYSTEM);
        }
    }
}

}

extern "C" void kernel_launch(int rank, int nranks, ncclComm_t comm,
                              hipStream_t stream,
                              void* const* d_in, const int* in_sizes, int n_in,
                              void* d_out, int out_size,
                              const int* in_dtypes, int out_dtype,
                              void* const* d_out_all,
                              void* const* d_ws_all, size_t ws_bytes) {
    (void)comm; (void)in_sizes; (void)n_in; (void)in_dtypes; (void)out_dtype; (void)d_out;
    if (nranks != NR || (unsigned)rank >= (unsigned)NR || ws_bytes < WS_NEED || out_size != MS * GN) {
        fprintf(stderr, "unsupported configuration\n");
        return;
    }
    static bool init = false;
    if (!init) {
        HIP_CHECK(hipFuncSetAttribute((const void*)fused_kernel, hipFuncAttributeMaxDynamicSharedMemorySize, LDS_BYTES));
        init = true;
    }
    Params P;
    P.x = (const uint8_t*)d_in[0];
    P.w = (const uint8_t*)d_in[1];
    P.sx = (const float*)d_in[2];
    P.sw = (const float*)d_in[3];
    for (int p = 0; p < NR; ++p) {
        P.out[p] = (float*)d_out_all[p];
        P.ws[p] = (uint8_t*)d_ws_all[p];
    }
    P.rank = rank;
    fused_kernel<<<GRID, THREADS, LDS_BYTES, stream>>>(P);
    HIP_CHECK(hipGetLastError());
}
```

```cpp
#include <hip/hip_runtime.h>
#include <rccl/rccl.h>
#include <cstdio>
#include <cstdint>

#define HIP_CHECK(expr) do { hipError_t _e = (expr); if (_e != hipSuccess) { \
    fprintf(stderr, "HIP error %s at %s:%d\n", hipGetErrorString(_e), __FILE__, __LINE__); \
    return; } } while (0)

namespace {

typedef int v8i __attribute__((ext_vector_type(8)));
typedef float v4f __attribute__((ext_vector_type(4)));
typedef unsigned v4u __attribute__((ext_vector_type(4)));
typedef __attribute__((address_space(3))) volatile unsigned lds_u32;

constexpr int NR = 8;
constexpr int GM = 4096, GK = 4096, GN = 16384;
constexpr int KS = GK / NR;
constexpr int NS = GN / NR;
constexpr int MS = GM / NR;
constexpr int TM = 128, TN = 256, BK = 128;
constexpr int NMB = GM / TM;
constexpr int NNB = NS / TN;
constexpr int NTILES = NMB * NNB;
constexpr int GTN = 128;
constexpr int NGT = NTILES * 2;
constexpr int NKT = GK / BK;

constexpr int GRID = 256, THREADS = 512;
constexpr int NCOMM = 32;
constexpr int NPOOL = GRID - NCOMM;

constexpr size_t XS_BYTES = (size_t)GM * GK;
constexpr size_t WT_BYTES = (size_t)NS * GK;
constexpr size_t STAGE_BYTES = XS_BYTES + WT_BYTES;
constexpr size_t OFF_Y = 2 * STAGE_BYTES;
constexpr size_t Y_BYTES = (size_t)NTILES * TM * TN * 4;
constexpr size_t OFF_CTL = OFF_Y + Y_BYTES;
constexpr size_t WS_NEED = OFF_CTL + 262144;
constexpr int CW = 32;
constexpr int C_CALL = 0 * CW, C_TMO = 2 * CW;
constexpr int C_XCNT = 8 * CW;
constexpr int C_WCNT = 264 * CW;
constexpr int C_YFLAG = 328 * CW;
constexpr int C_TILEQ = 840 * CW;
constexpr int C_DONE = 848 * CW;

constexpr int LW = NCOMM / 8;
constexpr int XU = 4;
constexpr int U_W0 = 8, U_X0 = U_W0 + 8 * XU, U_W1 = U_X0 + 56, U_XW = U_W1 + 24 * XU;
#define XW_U1 126
#define START_GATE 1
constexpr int U1 = XW_U1;
constexpr int NYU = 32 * 64;
constexpr unsigned XCNT_PER_CALL = XU;
constexpr unsigned WCNT_PER_CALL = 8;
#define Y_WAVES 3
constexpr int YW = Y_WAVES;

constexpr int STAGE_LDS = (GTN + TM) * BK;
constexpr int NSTG = 4;
constexpr int LDS_CTL = NSTG * STAGE_LDS;
constexpr int LDS_BYTES = LDS_CTL + 64;

constexpr unsigned long long DEADLINE_TICKS = 50000000ull;

struct Params {
    const uint8_t* x;
    const uint8_t* w;
    const float* sx;
    const float* sw;
    float* out[NR];
    uint8_t* ws[NR];
    int rank;
};

template <bool SYS, int SLP = 2>
__device__ __forceinline__ void wait_ge(unsigned* p, unsigned target, unsigned* tmo, unsigned call) {
    unsigned long long t0 = __builtin_amdgcn_s_memrealtime();
    for (;;) {
        unsigned v = SYS ? __hip_atomic_load(p, __ATOMIC_RELAXED, __HIP_MEMORY_SCOPE_SYSTEM)
                         : __hip_atomic_load(p, __ATOMIC_RELAXED, __HIP_MEMORY_SCOPE_AGENT);
        if (v >= target) break;
        if (__hip_atomic_load(tmo, __ATOMIC_RELAXED, __HIP_MEMORY_SCOPE_AGENT) == call ||
            __builtin_amdgcn_s_memrealtime() - t0 > DEADLINE_TICKS) {
            __hip_atomic_store(tmo, call, __ATOMIC_RELAXED, __HIP_MEMORY_SCOPE_AGENT);
            break;
        }
        __builtin_amdgcn_s_sleep(SLP);
    }
}

__device__ __forceinline__ void tr4x4(unsigned r0, unsigned r1, unsigned r2, unsigned r3,
                                      unsigned& o0, unsigned& o1, unsigned& o2, unsigned& o3) {
    unsigned t0 = __builtin_amdgcn_perm(r1, r0, 0x05010400u);
    unsigned t1 = __builtin_amdgcn_perm(r1, r0, 0x07030602u);
    unsigned t2 = __builtin_amdgcn_perm(r3, r2, 0x05010400u);
    unsigned t3 = __builtin_amdgcn_perm(r3, r2, 0x07030602u);
    o0 = __builtin_amdgcn_perm(t2, t0, 0x05040100u);
    o1 = __builtin_amdgcn_perm(t2, t0, 0x07060302u);
    o2 = __builtin_amdgcn_perm(t3, t1, 0x05040100u);
    o3 = __builtin_amdgcn_perm(t3, t1, 0x07060302u);
}

#define VMCNT0() asm volatile("s_waitcnt vmcnt(0)" ::: "memory")
__device__ __forceinline__ void aload(v4u& d, const uint8_t* p) {
    asm volatile("global_load_dwordx4 %0, %1, off" : "=v"(d) : "v"(p) : "memory");
}
__device__ __forceinline__ void aload_sys(v4u& d, const uint8_t* p) {
    asm volatile("global_load_dwordx4 %0, %1, off sc0 sc1" : "=v"(d) : "v"(p) : "memory");
}
#define TIE4(a, i) "+v"(a[i]), "+v"(a[i + 1]), "+v"(a[i + 2]), "+v"(a[i + 3])
#define WAIT16(n, a) asm volatile("s_waitcnt vmcnt(" #n ")" : TIE4(a, 0), TIE4(a, 4), TIE4(a, 8), TIE4(a, 12) :: "memory")
#define WAIT14(n, a) asm volatile("s_waitcnt vmcnt(" #n ")" : TIE4(a, 0), TIE4(a, 4), TIE4(a, 8), "+v"(a[12]), "+v"(a[13]) :: "memory")

__device__ __forceinline__ int y_grab(lds_u32* sctl, int lane) {
    unsigned j = 0;
    if (lane == 0) j = __hip_atomic_fetch_add((__attribute__((address_space(3))) unsigned*)&sctl[4], 1u,
                                              __ATOMIC_RELAXED, __HIP_MEMORY_SCOPE_WORKGROUP);
    return __builtin_amdgcn_readfirstlane((int)j);
}
__device__ __forceinline__ bool y_ready(int wg, int j, lds_u32* sctl) {
    return (unsigned)__builtin_amdgcn_readfirstlane(sctl[2]) > (unsigned)((j * NCOMM + wg) >> 6);
}
__device__ __forceinline__ void y_load(v4u (&in)[14], const Params& P, int rank, int wg, int j, int lane) {
    const int yu = j * NCOMM + wg;
    const int k = yu >> 6, sub = yu & 63;
    const size_t off = OFF_Y + (size_t)(rank * 32 + k) * (TM * TN * 4) + sub * 2048 + lane * 16;
#pragma unroll
    for (int h = 0; h < 2; ++h)
#pragma unroll
        for (int q = 0; q < 7; ++q)
            aload_sys(in[h * 7 + q], P.ws[(rank + 1 + q) & 7] + off + h * 1024);
}
__device__ __forceinline__ void y_store(v4u (&in)[14], const Params& P, int rank, int wg, int j, int lane,
                                        lds_u32* sctl) {
    const int yu = j * NCOMM + wg;
    const int k = yu >> 6, sub = yu & 63;
    const int rnd = k >> 3, nb = k & 7;
    const bool bad = __builtin_amdgcn_readfirstlane(sctl[3]) != 0;
    const v4u nanv = {0x7fc00000u, 0x7fc00000u, 0x7fc00000u, 0x7fc00000u};
    auto rs = __builtin_amdgcn_make_buffer_rsrc(P.out[rank], 0, (int)((size_t)MS * GN * 4), 0x00020000);
    const int voff = ((rnd * TM + sub * 2) * GN + nb * TN) * 4 + lane * 16;
#pragma unroll
    for (int h = 0; h < 2; ++h)
#pragma unroll
        for (int q = 0; q < 7; ++q)
            __builtin_amdgcn_raw_buffer_store_b128(bad ? nanv : in[h * 7 + q], rs,
                                                   voff + h * (GN * 4) + ((rank + 1 + q) & 7) * (NS * 4), 0, 16);
}

__device__ __forceinline__ void comm_role(const Params& P, unsigned call, int wg, int wave, int lane,
                                          lds_u32* sctl) {
    const int rank = P.rank;
    uint8_t* myws = P.ws[rank];
    unsigned* myctl = (unsigned*)(myws + OFF_CTL);
    unsigned* tmo = myctl + C_TMO;
    const size_t par = (size_t)(call & 1u) * STAGE_BYTES;
    if (wave == 7) {
        for (int k = 0; k < 32; ++k) {
            if (lane < 14) wait_ge<true>(myctl + C_YFLAG + ((((rank + 1 + (lane >> 1)) & 7) * 32 + k) * 2 + (lane & 1)) * CW, call, tmo, call);
            __builtin_amdgcn_fence(__ATOMIC_ACQUIRE, "");
            const bool bad = __hip_atomic_load(tmo, __ATOMIC_RELAXED, __HIP_MEMORY_SCOPE_AGENT) == call;
            VMCNT0();
            if (lane == 0) { if (bad) sctl[3] = 1u; sctl[2] = (unsigned)k + 1u; }
        }
        return;
    }
    const int JEND = (NYU - wg + NCOMM - 1) / NCOMM;
    v4u A[14], B[14];
    int j = 0;
    bool pre = false;
    {
        const int p = wg & 7;
        unsigned* dctl = (unsigned*)(P.ws[p] + OFF_CTL);
        auto rs = __builtin_amdgcn_make_buffer_rsrc(P.ws[p] + par, 0, (int)STAGE_BYTES, 0x00020000);
        int u = (wg >> 3) * 7 + wave;
        bool first = true;
        auto gate = [&]() {
            if (first && p != rank && lane == 0)
                wait_ge<true>(myctl + C_DONE + p * CW, (call - 1u) * (unsigned)(NCOMM * YW), tmo, call);
            first = false;
        };
        while (u < U_XW) {
            unsigned* cnt;
            if (u < U_W0 || (u >= U_X0 && u < U_W1)) {
                const int wu = (u < U_W0) ? u : (u - U_X0 + 8);
                const int nb = wu >> 3, kb = (wu >> 1) & 3, nh = wu & 1;
                const int kg = lane & 7, ng = lane >> 3;
                const uint8_t* src = P.w + (size_t)(kb * 128 + kg * 16) * GN + (p * NS + nb * TN + nh * 128 + ng * 16);
                v4u in[16];
#pragma unroll
                for (int i = 0; i < 16; ++i) aload(in[i], src + (size_t)i * GN);
                WAIT16(0, in);
                v4u o[16];
#pragma unroll
                for (int a = 0; a < 4; ++a)
#pragma unroll
                    for (int b = 0; b < 4; ++b) {
                        unsigned q0, q1, q2, q3;
                        tr4x4(in[4 * a + 0][b], in[4 * a + 1][b], in[4 * a + 2][b], in[4 * a + 3][b], q0, q1, q2, q3);
                        o[4 * b + 0][a] = q0; o[4 * b + 1][a] = q1; o[4 * b + 2][a] = q2; o[4 * b + 3][a] = q3;
                    }
                const int voff = (int)XS_BYTES + rank * (NS * KS) + (nb * TN + nh * 128 + ng * 16) * KS + kb * 128 + kg * 16;
                gate();
#pragma unroll
                for (int j = 0; j < 16; ++j)
                    __builtin_amdgcn_raw_buffer_store_b128(o[j], rs, voff + j * KS, 0, 17);
                cnt = dctl + C_WCNT + (nb * 8 + rank) * CW;
            } else {
                const int xu = (u < U_X0) ? (u - U_W0) : (u - U_W1 + 8 * XU);
                const int blk = xu / XU, sub = xu % XU;
                const int mb = ((p + 1 + (blk & 7)) & 7) * 4 + (blk >> 3);
                const int off = (mb * TM + sub * 32) * KS + lane * 16;
                v4u in[16];
#pragma unroll
                for (int i = 0; i < 16; ++i) in[i] = *(const v4u*)(P.x + off + i * 1024);
                gate();
#pragma unroll
                for (int i = 0; i < 16; ++i)
                    __builtin_amdgcn_raw_buffer_store_b128(in[i], rs, rank * (GM * KS) + off + i * 1024, 0, 17);
                cnt = dctl + C_XCNT + (mb * 8 + rank) * CW;
            }
            if (u + 7 * LW >= U1 && wave < YW) {
                j = y_grab(sctl, lane);
                pre = j < JEND && y_ready(wg, j, sctl);
                asm volatile("" ::: "memory");
                if (pre) y_load(A, P, rank, wg, j, lane);
                WAIT14(0, A);
            } else {
                VMCNT0();
            }
            if (lane == 0) __hip_atomic_fetch_add(cnt, 1u, __ATOMIC_RELAXED, __HIP_MEMORY_SCOPE_SYSTEM);
            if (u >= U1) u += (7 - YW) * LW;
            else if (u + 7 * LW < U1) u += 7 * LW;
            else u = (wave < YW) ? U_XW : U1 + (wg >> 3) * (7 - YW) + (wave - YW);
        }
    }
    if (wave < YW) {
        if (j < JEND) {
            if (!pre) {
                while (!y_ready(wg, j, sctl)) __builtin_amdgcn_s_sleep(1);
                asm volatile("" ::: "memory");
                y_load(A, P, rank, wg, j, lane);
                WAIT14(0, A);
            }
#define Y_HALF(CUR, NXT)                                                       \
            {                                                                  \
                const int j1 = y_grab(sctl, lane);                             \
                const bool more = j1 < JEND;                                   \
                const bool rdy = more && y_ready(wg, j1, sctl);                \
                asm volatile("" ::: "memory");                                 \
                if (rdy) y_load(NXT, P, rank, wg, j1, lane);                   \
                y_store(CUR, P, rank, wg, j, lane, sctl);                      \
                if (!more) break;                                              \
                if (rdy) { WAIT14(14, NXT); }                                  \
                else {                                                         \
                    while (!y_ready(wg, j1, sctl)) __builtin_amdgcn_s_sleep(1);\
                    asm volatile("" ::: "memory");                             \
                    y_load(NXT, P, rank, wg, j1, lane);                        \
                    WAIT14(0, NXT);                                            \
                }                                                              \
                j = j1;                                                        \
            }
            for (;;) {
                Y_HALF(A, B)
                Y_HALF(B, A)
            }
        }
        uint8_t* q = P.ws[0];
#pragma unroll
        for (int i = 1; i < NR; ++i) q = (((rank + 1 + lane) & 7) == i) ? P.ws[i] : q;
        if (lane < 7)
            __hip_atomic_fetch_add((unsigned*)(q + OFF_CTL) + C_DONE + rank * CW, 1u, __ATOMIC_RELAXED, __HIP_MEMORY_SCOPE_SYSTEM);
    }
}

__global__ __launch_bounds__(THREADS, 2) void fused_kernel(Params P) {
    extern __shared__ __attribute__((aligned(16))) uint8_t smem[];
    const int tid = threadIdx.x;
    const int wave = __builtin_amdgcn_readfirstlane(tid >> 6);
    const int lane = tid & 63;
    const int rank = P.rank;
    uint8_t* myws = P.ws[rank];
    unsigned* myctl = (unsigned*)(myws + OFF_CTL);
    unsigned* tmo = myctl + C_TMO;
    lds_u32* sctl = (lds_u32*)(smem + LDS_CTL);

    if (tid == 0) {
        unsigned t = __hip_atomic_fetch_add(myctl + C_CALL, 1u, __ATOMIC_RELAXED, __HIP_MEMORY_SCOPE_AGENT);
        sctl[0] = t / GRID + 1u;
        sctl[2] = 0u;
        sctl[3] = 0u;
        sctl[4] = 0u;
    }
    __syncthreads();
    const unsigned call = __builtin_amdgcn_readfirstlane(sctl[0]);

    const int wg = blockIdx.x;
    if (wg < NCOMM) {
        comm_role(P, call, wg, wave, lane, sctl);
        return;
    }

    const float scale = P.sx[0] * P.sw[0];
    const uint8_t* stage = myws + (size_t)(call & 1u) * STAGE_BYTES;
    const int wn = wave >> 2, wm = wave & 3;
    const int fr = lane & 15, fq = lane >> 4;
    const int sw0 = ((fq) ^ (fr & 7)) * 16;
    const int a_off = (wn * 64 + fr) * BK + sw0;
    const int b_off = GTN * BK + (wm * 32 + fr) * BK + sw0;
    const int srow = lane >> 3;
    const int schunk = ((lane & 7) ^ (srow & 7)) * 16;

    for (;;) {
        __syncthreads();
        if (wave == 0) {
            unsigned t = 0;
            if (lane == 0) {
                t = __hip_atomic_fetch_add(myctl + C_TILEQ + (wg & 7) * CW, 1u, __ATOMIC_RELAXED, __HIP_MEMORY_SCOPE_AGENT);
                t -= (call - 1u) * (unsigned)((NGT + NPOOL) / 8);
                if (t < (unsigned)(NGT / 8)) {
                    const unsigned rnd_ = t >> 4, oi_ = (t >> 1) & 7, nb_ = wg & 7;
                    const unsigned t2_ = rnd_ * 64 + (rnd_ == 0 ? nb_ * 8 + oi_ : oi_ * 8 + nb_);
                    t = t2_ * 2 + (t & 1);
                } else {
                    t = NGT;
                }
                sctl[1] = t;
            }
            t = __builtin_amdgcn_readfirstlane(t);
            if (t < (unsigned)NGT) {
                const int t2 = t >> 1;
                const int nb = (t2 < 64) ? (t2 >> 3) : (t2 & 7), oi = (t2 < 64) ? (t2 & 7) : ((t2 >> 3) & 7);
                const int mb = ((rank + 1 + oi) & 7) * 4 + (t2 >> 6);
                if (lane < 16)
                    wait_ge<true, 16>(myctl + ((lane < 8) ? C_XCNT + (mb * 8 + lane) * CW : C_WCNT + (nb * 8 + lane - 8) * CW),
                                  call * ((lane < 8) ? XCNT_PER_CALL : WCNT_PER_CALL), tmo, call);
                __builtin_amdgcn_fence(__ATOMIC_ACQUIRE, "");
                VMCNT0();
            }
        }
        __syncthreads();
        const unsigned t = __builtin_amdgcn_readfirstlane(sctl[1]);
        if (t >= (unsigned)NGT) break;
        const int t2 = t >> 1, half = t & 1;
        const int rnd = t2 >> 6, nb = (t2 < 64) ? (t2 >> 3) : (t2 & 7), oi = (t2 < 64) ? (t2 & 7) : ((t2 >> 3) & 7);
        const int owner = (rank + 1 + oi) & 7;
        const int mb = owner * 4 + rnd;

        const uint8_t* gA = stage + XS_BYTES + (size_t)(nb * TN + half * GTN + wave * 16 + srow) * KS + schunk;
        const uint8_t* gB = stage + (size_t)(mb * TM + wave * 16 + srow) * KS + schunk;
        auto issue = [&](int kt, int buf) {
            uint8_t* la = smem + buf + wave * 2048;
            uint8_t* lb = smem + buf + GTN * BK + wave * 2048;
            const uint8_t* a = gA + (size_t)(kt >> 2) * (NS * KS) + (kt & 3) * BK;
            const uint8_t* b = gB + (size_t)(kt >> 2) * (GM * KS) + (kt & 3) * BK;
#pragma unroll
            for (int j = 0; j < 2; ++j)
                __builtin_amdgcn_global_load_lds((const unsigned*)(a + j * 8 * KS), (unsigned*)(la + j * 1024), 16, 0, 0);
#pragma unroll
            for (int j = 0; j < 2; ++j)
                __builtin_amdgcn_global_load_lds((const unsigned*)(b + j * 8 * KS), (unsigned*)(lb + j * 1024), 16, 0, 0);
        };

        v4f acc[4][2];
#pragma unroll
        for (int i = 0; i < 4; ++i)
#pragma unroll
            for (int j = 0; j < 2; ++j) acc[i][j] = (v4f){0.f, 0.f, 0.f, 0.f};

        issue(0, 0);
        issue(1, STAGE_LDS);
        issue(2, 2 * STAGE_LDS);
        for (int kt = 0; kt < NKT; ++kt) {
            if (kt < NKT - 2) asm volatile("s_waitcnt vmcnt(8)" ::: "memory");
            else if (kt == NKT - 2) asm volatile("s_waitcnt vmcnt(4)" ::: "memory");
            else asm volatile("s_waitcnt vmcnt(0)" ::: "memory");
            asm volatile("s_waitcnt lgkmcnt(0)" ::: "memory");
            __builtin_amdgcn_s_barrier();
            asm volatile("" ::: "memory");
            const int cur = (kt & 3) * STAGE_LDS;
            if (kt + 3 < NKT) issue(kt + 3, ((kt + 3) & 3) * STAGE_LDS);
            v8i af[4], bf[2];
#pragma unroll
            for (int i = 0; i < 4; ++i) {
                const v4u lo = *(const v4u*)(smem + cur + a_off + i * 2048);
                const v4u hi = *(const v4u*)(smem + cur + (a_off ^ 64) + i * 2048);
                af[i] = (v8i){(int)lo[0], (int)lo[1], (int)lo[2], (int)lo[3], (int)hi[0], (int)hi[1], (int)hi[2], (int)hi[3]};
            }
#pragma unroll
            for (int i = 0; i < 2; ++i) {
                const v4u lo = *(const v4u*)(smem + cur + b_off + i * 2048);
                const v4u hi = *(const v4u*)(smem + cur + (b_off ^ 64) + i * 2048);
                bf[i] = (v8i){(int)lo[0], (int)lo[1], (int)lo[2], (int)lo[3], (int)hi[0], (int)hi[1], (int)hi[2], (int)hi[3]};
            }
#pragma unroll
            for (int i = 0; i < 4; ++i)
#pragma unroll
                for (int j = 0; j < 2; ++j)
                    acc[i][j] = __builtin_amdgcn_mfma_scale_f32_16x16x128_f8f6f4(af[i], bf[j], acc[i][j], 0, 0, 0,
                                                                                 0x7F7F7F7F, 0, 0x7F7F7F7F);
        }

        const bool bad = __hip_atomic_load(tmo, __ATOMIC_RELAXED, __HIP_MEMORY_SCOPE_AGENT) == call;
        const float s = bad ? __builtin_nanf("") : scale;
        if (owner == rank) {
            auto rs_o = __builtin_amdgcn_make_buffer_rsrc(P.out[rank], 0, (int)((size_t)MS * GN * 4), 0x00020000);
            const int voff = ((rnd * TM + wm * 32 + fr) * GN + rank * NS + nb * TN + half * GTN + wn * 64 + fq * 4) * 4;
#pragma unroll
            for (int j = 0; j < 2; ++j)
#pragma unroll
                for (int i = 0; i < 4; ++i) {
                    v4f v = acc[i][j] * s;
                    __builtin_amdgcn_raw_buffer_store_b128(__builtin_bit_cast(v4u, v), rs_o, voff + (j * 16 * GN + i * 16) * 4, 0, 16);
                }
        } else {
            const int tile = mb * NNB + nb;
            auto rs_y = __builtin_amdgcn_make_buffer_rsrc(myws + OFF_Y + (size_t)tile * (TM * TN * 4), 0, TM * TN * 4, 0x00020000);
            const int voff = ((wm * 32 + fr) * TN + half * GTN + wn * 64 + fq * 4) * 4;
#pragma unroll
            for (int j = 0; j < 2; ++j)
#pragma unroll
                for (int i = 0; i < 4; ++i) {
                    v4f v = acc[i][j] * s;
                    __builtin_amdgcn_raw_buffer_store_b128(__builtin_bit_cast(v4u, v), rs_y, voff + (j * 16 * TN + i * 16) * 4, 0, 17);
                }
            VMCNT0();
            __syncthreads();
            if (tid == 0)
                __hip_atomic_store((unsigned*)(P.ws[owner] + OFF_CTL) + C_YFLAG + ((rank * 32 + rnd * 8 + nb) * 2 + half) * CW,
                                   call, __ATOMIC_RELAXED, __HIP_MEMORY_SCOPE_SYSTEM);
        }
    }
}

}

extern "C" void kernel_launch(int rank, int nranks, ncclComm_t comm,
                              hipStream_t stream,
                              void* const* d_in, const int* in_sizes, int n_in,
                              void* d_out, int out_size,
                              const int* in_dtypes, int out_dtype,
                              void* const* d_out_all,
                              void* const* d_ws_all, size_t ws_bytes) {
    (void)comm; (void)in_sizes; (void)n_in; (void)in_dtypes; (void)out_dtype; (void)d_out;
    if (nranks != NR || (unsigned)rank >= (unsigned)NR || ws_bytes < WS_NEED || out_size != MS * GN) {
        fprintf(stderr, "unsupported configuration\n");
        return;
    }
    static bool init = false;
    if (!init) {
        HIP_CHECK(hipFuncSetAttribute((const void*)fused_kernel, hipFuncAttributeMaxDynamicSharedMemorySize, LDS_BYTES));
        init = true;
    }
    Params P;
    P.x = (const uint8_t*)d_in[0];
    P.w = (const uint8_t*)d_in[1];
    P.sx = (const float*)d_in[2];
    P.sw = (const float*)d_in[3];
    for (int p = 0; p < NR; ++p) {
        P.out[p] = (float*)d_out_all[p];
        P.ws[p] = (uint8_t*)d_ws_all[p];
    }
    P.rank = rank;
    fused_kernel<<<GRID, THREADS, LDS_BYTES, stream>>>(P);
    HIP_CHECK(hipGetLastError());
}
```

```cpp
#include <hip/hip_runtime.h>
#include <rccl/rccl.h>
#include <cstdio>
#include <cstdint>

#define HIP_CHECK(expr) do { hipError_t _e = (expr); if (_e != hipSuccess) { \
    fprintf(stderr, "HIP error %s at %s:%d\n", hipGetErrorString(_e), __FILE__, __LINE__); \
    return; } } while (0)

namespace {

typedef int v8i __attribute__((ext_vector_type(8)));
typedef float v4f __attribute__((ext_vector_type(4)));
typedef unsigned v4u __attribute__((ext_vector_type(4)));
typedef __attribute__((address_space(3))) volatile unsigned lds_u32;

constexpr int NR = 8;
constexpr int GM = 4096, GK = 4096, GN = 16384;
constexpr int KS = GK / NR;
constexpr int NS = GN / NR;
constexpr int MS = GM / NR;
constexpr int TM = 128, TN = 256, BK = 128;
constexpr int NMB = GM / TM;
constexpr int NNB = NS / TN;
constexpr int NTILES = NMB * NNB;
constexpr int GTN = 128;
constexpr int NGT = NTILES * 2;
constexpr int NKT = GK / BK;

constexpr int GRID = 256, THREADS = 512;
constexpr int NCOMM = 32;
constexpr int NPOOL = GRID - NCOMM;

constexpr size_t XS_BYTES = (size_t)GM * GK;
constexpr size_t WT_BYTES = (size_t)NS * GK;
constexpr size_t STAGE_BYTES = XS_BYTES + WT_BYTES;
constexpr size_t OFF_Y = 2 * STAGE_BYTES;
constexpr size_t Y_BYTES = (size_t)NTILES * TM * TN * 4;
constexpr size_t OFF_CTL = OFF_Y + Y_BYTES;
constexpr size_t WS_NEED = OFF_CTL + 262144;
constexpr int CW = 32;
constexpr int C_CALL = 0 * CW, C_TMO = 2 * CW;
constexpr int C_XCNT = 8 * CW;
constexpr int C_WCNT = 264 * CW;
constexpr int C_YFLAG = 328 * CW;
constexpr int C_TILEQ = 840 * CW;
constexpr int C_DONE = 848 * CW;

constexpr int LW = NCOMM / 8;
constexpr int XU = 4;
constexpr int U_W0 = 8, U_X0 = U_W0 + 8 * XU, U_W1 = U_X0 + 56, U_XW = U_W1 + 24 * XU;
constexpr int U1 = 126;
constexpr int NYU = 32 * 64;
constexpr unsigned XCNT_PER_CALL = XU;
constexpr unsigned WCNT_PER_CALL = 8;
constexpr int YW = 3;

constexpr int STAGE_LDS = (GTN + TM) * BK;
constexpr int NSTG = 4;
constexpr int LDS_CTL = NSTG * STAGE_LDS;
constexpr int LDS_BYTES = LDS_CTL + 64;

constexpr unsigned long long DEADLINE_TICKS = 50000000ull;

struct Params {
    const uint8_t* x;
    const uint8_t* w;
    const float* sx;
    const float* sw;
    float* out[NR];
    uint8_t* ws[NR];
    int rank;
};

template <bool SYS, int SLP = 2>
__device__ __forceinline__ void wait_ge(unsigned* p, unsigned target, unsigned* tmo, unsigned call) {
    unsigned long long t0 = __builtin_amdgcn_s_memrealtime();
    for (;;) {
        unsigned v = SYS ? __hip_atomic_load(p, __ATOMIC_RELAXED, __HIP_MEMORY_SCOPE_SYSTEM)
                         : __hip_atomic_load(p, __ATOMIC_RELAXED, __HIP_MEMORY_SCOPE_AGENT);
        if (v >= target) break;
        if (__hip_atomic_load(tmo, __ATOMIC_RELAXED, __HIP_MEMORY_SCOPE_AGENT) == call ||
            __builtin_amdgcn_s_memrealtime() - t0 > DEADLINE_TICKS) {
            __hip_atomic_store(tmo, call, __ATOMIC_RELAXED, __HIP_MEMORY_SCOPE_AGENT);
            break;
        }
        __builtin_amdgcn_s_sleep(SLP);
    }
}

__device__ __forceinline__ void tr4x4(unsigned r0, unsigned r1, unsigned r2, unsigned r3,
                                      unsigned& o0, unsigned& o1, unsigned& o2, unsigned& o3) {
    unsigned t0 = __builtin_amdgcn_perm(r1, r0, 0x05010400u);
    unsigned t1 = __builtin_amdgcn_perm(r1, r0, 0x07030602u);
    unsigned t2 = __builtin_amdgcn_perm(r3, r2, 0x05010400u);
    unsigned t3 = __builtin_amdgcn_perm(r3, r2, 0x07030602u);
    o0 = __builtin_amdgcn_perm(t2, t0, 0x05040100u);
    o1 = __builtin_amdgcn_perm(t2, t0, 0x07060302u);
    o2 = __builtin_amdgcn_perm(t3, t1, 0x05040100u);
    o3 = __builtin_amdgcn_perm(t3, t1, 0x07060302u);
}

#define VMCNT0() asm volatile("s_waitcnt vmcnt(0)" ::: "memory")
__device__ __forceinline__ void aload(v4u& d, const uint8_t* p) {
    asm volatile("global_load_dwordx4 %0, %1, off" : "=v"(d) : "v"(p) : "memory");
}
__device__ __forceinline__ void aload_sys(v4u& d, const uint8_t* p) {
    asm volatile("global_load_dwordx4 %0, %1, off sc0 sc1" : "=v"(d) : "v"(p) : "memory");
}
#define TIE4(a, i) "+v"(a[i]), "+v"(a[i + 1]), "+v"(a[i + 2]), "+v"(a[i + 3])
#define WAIT16(n, a) asm volatile("s_waitcnt vmcnt(" #n ")" : TIE4(a, 0), TIE4(a, 4), TIE4(a, 8), TIE4(a, 12) :: "memory")
#define WAIT14(n, a) asm volatile("s_waitcnt vmcnt(" #n ")" : TIE4(a, 0), TIE4(a, 4), TIE4(a, 8), "+v"(a[12]), "+v"(a[13]) :: "memory")

__device__ __forceinline__ int y_grab(lds_u32* sctl, int lane) {
    unsigned j = 0;
    if (lane == 0) j = __hip_atomic_fetch_add((__attribute__((address_space(3))) unsigned*)&sctl[4], 1u,
                                              __ATOMIC_RELAXED, __HIP_MEMORY_SCOPE_WORKGROUP);
    return __builtin_amdgcn_readfirstlane((int)j);
}
__device__ __forceinline__ bool y_ready(int wg, int j, lds_u32* sctl) {
    return (unsigned)__builtin_amdgcn_readfirstlane(sctl[2]) > (unsigned)((j * NCOMM + wg) >> 6);
}
__device__ __forceinline__ void y_load(v4u (&in)[14], const Params& P, int rank, int wg, int j, int lane) {
    const int yu = j * NCOMM + wg;
    const int k = yu >> 6, sub = yu & 63;
    const size_t off = OFF_Y + (size_t)(rank * 32 + k) * (TM * TN * 4) + sub * 2048 + lane * 16;
#pragma unroll
    for (int h = 0; h < 2; ++h)
#pragma unroll
        for (int q = 0; q < 7; ++q)
            aload_sys(in[h * 7 + q], P.ws[(rank + 1 + q) & 7] + off + h * 1024);
}
__device__ __forceinline__ void y_store(v4u (&in)[14], const Params& P, int rank, int wg, int j, int lane,
                                        lds_u32* sctl) {
    const int yu = j * NCOMM + wg;
    const int k = yu >> 6, sub = yu & 63;
    const int rnd = k >> 3, nb = k & 7;
    const bool bad = __builtin_amdgcn_readfirstlane(sctl[3]) != 0;
    const v4u nanv = {0x7fc00000u, 0x7fc00000u, 0x7fc00000u, 0x7fc00000u};
    auto rs = __builtin_amdgcn_make_buffer_rsrc(P.out[rank], 0, (int)((size_t)MS * GN * 4), 0x00020000);
    const int voff = ((rnd * TM + sub * 2) * GN + nb * TN) * 4 + lane * 16;
#pragma unroll
    for (int h = 0; h < 2; ++h)
#pragma unroll
        for (int q = 0; q < 7; ++q)
            __builtin_amdgcn_raw_buffer_store_b128(bad ? nanv : in[h * 7 + q], rs,
                                                   voff + h * (GN * 4) + ((rank + 1 + q) & 7) * (NS * 4), 0, 16);
}

__device__ __forceinline__ void comm_role(const Params& P, unsigned call, int wg, int wave, int lane,
                                          lds_u32* sctl) {
    const int rank = P.rank;
    uint8_t* myws = P.ws[rank];
    unsigned* myctl = (unsigned*)(myws + OFF_CTL);
    unsigned* tmo = myctl + C_TMO;
    const size_t par = (size_t)(call & 1u) * STAGE_BYTES;
    if (wave == 7) {
        for (int k = 0; k < 32; ++k) {
            if (lane < 14) wait_ge<true>(myctl + C_YFLAG + ((((rank + 1 + (lane >> 1)) & 7) * 32 + k) * 2 + (lane & 1)) * CW, call, tmo, call);
            __builtin_amdgcn_fence(__ATOMIC_ACQUIRE, "");
            const bool bad = __hip_atomic_load(tmo, __ATOMIC_RELAXED, __HIP_MEMORY_SCOPE_AGENT) == call;
            VMCNT0();
            if (lane == 0) { if (bad) sctl[3] = 1u; sctl[2] = (unsigned)k + 1u; }
        }
        return;
    }
    const int JEND = (NYU - wg + NCOMM - 1) / NCOMM;
    v4u A[14], B[14];
    int j = 0;
    bool pre = false;
    {
        const int p = wg & 7;
        unsigned* dctl = (unsigned*)(P.ws[p] + OFF_CTL);
        auto rs = __builtin_amdgcn_make_buffer_rsrc(P.ws[p] + par, 0, (int)STAGE_BYTES, 0x00020000);
        int u = (wg >> 3) * 7 + wave;
        bool first = true;
        auto gate = [&]() {
            if (first && p != rank && lane == 0)
                wait_ge<true>(myctl + C_DONE + p * CW, (call - 1u) * (unsigned)(NCOMM * YW), tmo, call);
            first = false;
        };
        while (u < U_XW) {
            unsigned* cnt;
            if (u < U_W0 || (u >= U_X0 && u < U_W1)) {
                const int wu = (u < U_W0) ? u : (u - U_X0 + 8);
                const int nb = wu >> 3, kb = (wu >> 1) & 3, nh = wu & 1;
                const int kg = lane & 7, ng = lane >> 3;
                const uint8_t* src = P.w + (size_t)(kb * 128 + kg * 16) * GN + (p * NS + nb * TN + nh * 128 + ng * 16);
                v4u in[16];
#pragma unroll
                for (int i = 0; i < 16; ++i) aload(in[i], src + (size_t)i * GN);
                WAIT16(0, in);
                v4u o[16];
#pragma unroll
                for (int a = 0; a < 4; ++a)
#pragma unroll
                    for (int b = 0; b < 4; ++b) {
                        unsigned q0, q1, q2, q3;
                        tr4x4(in[4 * a + 0][b], in[4 * a + 1][b], in[4 * a + 2][b], in[4 * a + 3][b], q0, q1, q2, q3);
                        o[4 * b + 0][a] = q0; o[4 * b + 1][a] = q1; o[4 * b + 2][a] = q2; o[4 * b + 3][a] = q3;
                    }
                const int voff = (int)XS_BYTES + rank * (NS * KS) + (nb * TN + nh * 128 + ng * 16) * KS + kb * 128 + kg * 16;
                gate();
#pragma unroll
                for (int j = 0; j < 16; ++j)
                    __builtin_amdgcn_raw_buffer_store_b128(o[j], rs, voff + j * KS, 0, 17);
                cnt = dctl + C_WCNT + (nb * 8 + rank) * CW;
            } else {
                const int xu = (u < U_X0) ? (u - U_W0) : (u - U_W1 + 8 * XU);
                const int blk = xu / XU, sub = xu % XU;
                const int mb = ((p + 1 + (blk & 7)) & 7) * 4 + (blk >> 3);
                const int off = (mb * TM + sub * 32) * KS + lane * 16;
                v4u in[16];
#pragma unroll
                for (int i = 0; i < 16; ++i) in[i] = *(const v4u*)(P.x + off + i * 1024);
                gate();
#pragma unroll
                for (int i = 0; i < 16; ++i)
                    __builtin_amdgcn_raw_buffer_store_b128(in[i], rs, rank * (GM * KS) + off + i * 1024, 0, 17);
                cnt = dctl + C_XCNT + (mb * 8 + rank) * CW;
            }
            if (u + 7 * LW >= U1 && wave < YW) {
                j = y_grab(sctl, lane);
                pre = j < JEND && y_ready(wg, j, sctl);
                asm volatile("" ::: "memory");
                if (pre) y_load(A, P, rank, wg, j, lane);
                WAIT14(0, A);
            } else {
                VMCNT0();
            }
            if (lane == 0) __hip_atomic_fetch_add(cnt, 1u, __ATOMIC_RELAXED, __HIP_MEMORY_SCOPE_SYSTEM);
            if (u >= U1) u += (7 - YW) * LW;
            else if (u + 7 * LW < U1) u += 7 * LW;
            else u = (wave < YW) ? U_XW : U1 + (wg >> 3) * (7 - YW) + (wave - YW);
        }
    }
    if (wave < YW) {
        if (j < JEND) {
            if (!pre) {
                while (!y_ready(wg, j, sctl)) __builtin_amdgcn_s_sleep(1);
                asm volatile("" ::: "memory");
                y_load(A, P, rank, wg, j, lane);
                WAIT14(0, A);
            }
#define Y_HALF(CUR, NXT)                                                       \
            {                                                                  \
                const int j1 = y_grab(sctl, lane);                             \
                const bool more = j1 < JEND;                                   \
                const bool rdy = more && y_ready(wg, j1, sctl);                \
                asm volatile("" ::: "memory");                                 \
                if (rdy) y_load(NXT, P, rank, wg, j1, lane);                   \
                y_store(CUR, P, rank, wg, j, lane, sctl);                      \
                if (!more) break;                                              \
                if (rdy) { WAIT14(14, NXT); }                                  \
                else {                                                         \
                    while (!y_ready(wg, j1, sctl)) __builtin_amdgcn_s_sleep(1);\
                    asm volatile("" ::: "memory");                             \
                    y_load(NXT, P, rank, wg, j1, lane);                        \
                    WAIT14(0, NXT);                                            \
                }                                                              \
                j = j1;                                                        \
            }
            for (;;) {
                Y_HALF(A, B)
                Y_HALF(B, A)
            }
        }
        uint8_t* q = P.ws[0];
#pragma unroll
        for (int i = 1; i < NR; ++i) q = (((rank + 1 + lane) & 7) == i) ? P.ws[i] : q;
        if (lane < 7)
            __hip_atomic_fetch_add((unsigned*)(q + OFF_CTL) + C_DONE + rank * CW, 1u, __ATOMIC_RELAXED, __HIP_MEMORY_SCOPE_SYSTEM);
    }
}

__global__ __launch_bounds__(THREADS, 2) void fused_kernel(Params P) {
    extern __shared__ __attribute__((aligned(16))) uint8_t smem[];
    const int tid = threadIdx.x;
    const int wave = __builtin_amdgcn_readfirstlane(tid >> 6);
    const int lane = tid & 63;
    const int rank = P.rank;
    uint8_t* myws = P.ws[rank];
    unsigned* myctl = (unsigned*)(myws + OFF_CTL);
    unsigned* tmo = myctl + C_TMO;
    lds_u32* sctl = (lds_u32*)(smem + LDS_CTL);

    if (tid == 0) {
        unsigned t = __hip_atomic_fetch_add(myctl + C_CALL, 1u, __ATOMIC_RELAXED, __HIP_MEMORY_SCOPE_AGENT);
        sctl[0] = t / GRID + 1u;
        sctl[2] = 0u;
        sctl[3] = 0u;
        sctl[4] = 0u;
    }
    __syncthreads();
    const unsigned call = __builtin_amdgcn_readfirstlane(sctl[0]);

    const int wg = blockIdx.x;
    if (wg < NCOMM) {
        comm_role(P, call, wg, wave, lane, sctl);
        return;
    }

    const float scale = P.sx[0] * P.sw[0];
    const uint8_t* stage = myws + (size_t)(call & 1u) * STAGE_BYTES;
    const int wn = wave >> 2, wm = wave & 3;
    const int fr = lane & 15, fq = lane >> 4;
    const int sw0 = ((fq) ^ (fr & 7)) * 16;
    const int a_off = (wn * 64 + fr) * BK + sw0;
    const int b_off = GTN * BK + (wm * 32 + fr) * BK + sw0;
    const int srow = lane >> 3;
    const int schunk = ((lane & 7) ^ (srow & 7)) * 16;

    for (;;) {
        __syncthreads();
        if (wave == 0) {
            unsigned t = 0;
            if (lane == 0) {
                t = __hip_atomic_fetch_add(myctl + C_TILEQ + (wg & 7) * CW, 1u, __ATOMIC_RELAXED, __HIP_MEMORY_SCOPE_AGENT);
                t -= (call - 1u) * (unsigned)((NGT + NPOOL) / 8);
                if (t < (unsigned)(NGT / 8)) {
                    const unsigned rnd_ = t >> 4, oi_ = (t >> 1) & 7, nb_ = wg & 7;
                    const unsigned t2_ = rnd_ * 64 + (rnd_ == 0 ? nb_ * 8 + oi_ : oi_ * 8 + nb_);
                    t = t2_ * 2 + (t & 1);
                } else {
                    t = NGT;
                }
                sctl[1] = t;
            }
            t = __builtin_amdgcn_readfirstlane(t);
            if (t < (unsigned)NGT) {
                const int t2 = t >> 1;
                const int nb = (t2 < 64) ? (t2 >> 3) : (t2 & 7), oi = (t2 < 64) ? (t2 & 7) : ((t2 >> 3) & 7);
                const int mb = ((rank + 1 + oi) & 7) * 4 + (t2 >> 6);
                if (lane < 16)
                    wait_ge<true, 16>(myctl + ((lane < 8) ? C_XCNT + (mb * 8 + lane) * CW : C_WCNT + (nb * 8 + lane - 8) * CW),
                                  call * ((lane < 8) ? XCNT_PER_CALL : WCNT_PER_CALL), tmo, call);
                __builtin_amdgcn_fence(__ATOMIC_ACQUIRE, "");
                VMCNT0();
            }
        }
        __syncthreads();
        const unsigned t = __builtin_amdgcn_readfirstlane(sctl[1]);
        if (t >= (unsigned)NGT) break;
        const int t2 = t >> 1, half = t & 1;
        const int rnd = t2 >> 6, nb = (t2 < 64) ? (t2 >> 3) : (t2 & 7), oi = (t2 < 64) ? (t2 & 7) : ((t2 >> 3) & 7);
        const int owner = (rank + 1 + oi) & 7;
        const int mb = owner * 4 + rnd;

        const uint8_t* gA = stage + XS_BYTES + (size_t)(nb * TN + half * GTN + wave * 16 + srow) * KS + schunk;
        const uint8_t* gB = stage + (size_t)(mb * TM + wave * 16 + srow) * KS + schunk;
        auto issue = [&](int kt, int buf) {
            uint8_t* la = smem + buf + wave * 2048;
            uint8_t* lb = smem + buf + GTN * BK + wave * 2048;
            const uint8_t* a = gA + (size_t)(kt >> 2) * (NS * KS) + (kt & 3) * BK;
            const uint8_t* b = gB + (size_t)(kt >> 2) * (GM * KS) + (kt & 3) * BK;
#pragma unroll
            for (int j = 0; j < 2; ++j)
                __builtin_amdgcn_global_load_lds((const unsigned*)(a + j * 8 * KS), (unsigned*)(la + j * 1024), 16, 0, 0);
#pragma unroll
            for (int j = 0; j < 2; ++j)
                __builtin_amdgcn_global_load_lds((const unsigned*)(b + j * 8 * KS), (unsigned*)(lb + j * 1024), 16, 0, 0);
        };

        v4f acc[4][2];
#pragma unroll
        for (int i = 0; i < 4; ++i)
#pragma unroll
            for (int j = 0; j < 2; ++j) acc[i][j] = (v4f){0.f, 0.f, 0.f, 0.f};

        issue(0, 0);
        issue(1, STAGE_LDS);
        issue(2, 2 * STAGE_LDS);
        for (int kt = 0; kt < NKT; ++kt) {
            if (kt < NKT - 2) asm volatile("s_waitcnt vmcnt(8)" ::: "memory");
            else if (kt == NKT - 2) asm volatile("s_waitcnt vmcnt(4)" ::: "memory");
            else asm volatile("s_waitcnt vmcnt(0)" ::: "memory");
            asm volatile("s_waitcnt lgkmcnt(0)" ::: "memory");
            __builtin_amdgcn_s_barrier();
            asm volatile("" ::: "memory");
            const int cur = (kt & 3) * STAGE_LDS;
            if (kt + 3 < NKT) issue(kt + 3, ((kt + 3) & 3) * STAGE_LDS);
            v8i af[4], bf[2];
#pragma unroll
            for (int i = 0; i < 4; ++i) {
                const v4u lo = *(const v4u*)(smem + cur + a_off + i * 2048);
                const v4u hi = *(const v4u*)(smem + cur + (a_off ^ 64) + i * 2048);
                af[i] = (v8i){(int)lo[0], (int)lo[1], (int)lo[2], (int)lo[3], (int)hi[0], (int)hi[1], (int)hi[2], (int)hi[3]};
            }
#pragma unroll
            for (int i = 0; i < 2; ++i) {
                const v4u lo = *(const v4u*)(smem + cur + b_off + i * 2048);
                const v4u hi = *(const v4u*)(smem + cur + (b_off ^ 64) + i * 2048);
                bf[i] = (v8i){(int)lo[0], (int)lo[1], (int)lo[2], (int)lo[3], (int)hi[0], (int)hi[1], (int)hi[2], (int)hi[3]};
            }
#pragma unroll
            for (int i = 0; i < 4; ++i)
#pragma unroll
                for (int j = 0; j < 2; ++j)
                    acc[i][j] = __builtin_amdgcn_mfma_scale_f32_16x16x128_f8f6f4(af[i], bf[j], acc[i][j], 0, 0, 0,
                                                                                 0x7F7F7F7F, 0, 0x7F7F7F7F);
        }

        const bool bad = __hip_atomic_load(tmo, __ATOMIC_RELAXED, __HIP_MEMORY_SCOPE_AGENT) == call;
        const float s = bad ? __builtin_nanf("") : scale;
        if (owner == rank) {
            auto rs_o = __builtin_amdgcn_make_buffer_rsrc(P.out[rank], 0, (int)((size_t)MS * GN * 4), 0x00020000);
            const int voff = ((rnd * TM + wm * 32 + fr) * GN + rank * NS + nb * TN + half * GTN + wn * 64 + fq * 4) * 4;
#pragma unroll
            for (int j = 0; j < 2; ++j)
#pragma unroll
                for (int i = 0; i < 4; ++i) {
                    v4f v = acc[i][j] * s;
                    __builtin_amdgcn_raw_buffer_store_b128(__builtin_bit_cast(v4u, v), rs_o, voff + (j * 16 * GN + i * 16) * 4, 0, 16);
                }
        } else {
            const int tile = mb * NNB + nb;
            auto rs_y = __builtin_amdgcn_make_buffer_rsrc(myws + OFF_Y + (size_t)tile * (TM * TN * 4), 0, TM * TN * 4, 0x00020000);
            const int voff = ((wm * 32 + fr) * TN + half * GTN + wn * 64 + fq * 4) * 4;
#pragma unroll
            for (int j = 0; j < 2; ++j)
#pragma unroll
                for (int i = 0; i < 4; ++i) {
                    v4f v = acc[i][j] * s;
                    __builtin_amdgcn_raw_buffer_store_b128(__builtin_bit_cast(v4u, v), rs_y, voff + (j * 16 * TN + i * 16) * 4, 0, 17);
                }
            VMCNT0();
            __syncthreads();
            if (tid == 0)
                __hip_atomic_store((unsigned*)(P.ws[owner] + OFF_CTL) + C_YFLAG + ((rank * 32 + rnd * 8 + nb) * 2 + half) * CW,
                                   call, __ATOMIC_RELAXED, __HIP_MEMORY_SCOPE_SYSTEM);
        }
    }
}

}

extern "C" void kernel_launch(int rank, int nranks, ncclComm_t comm,
                              hipStream_t stream,
                              void* const* d_in, const int* in_sizes, int n_in,
                              void* d_out, int out_size,
                              const int* in_dtypes, int out_dtype,
                              void* const* d_out_all,
                              void* const* d_ws_all, size_t ws_bytes) {
    (void)comm; (void)in_sizes; (void)n_in; (void)in_dtypes; (void)out_dtype; (void)d_out;
    if (nranks != NR || (unsigned)rank >= (unsigned)NR || ws_bytes < WS_NEED || out_size != MS * GN) {
        fprintf(stderr, "unsupported configuration\n");
        return;
    }
    static bool init = false;
    if (!init) {
        HIP_CHECK(hipFuncSetAttribute((const void*)fused_kernel, hipFuncAttributeMaxDynamicSharedMemorySize, LDS_BYTES));
        init = true;
    }
    Params P;
    P.x = (const uint8_t*)d_in[0];
    P.w = (const uint8_t*)d_in[1];
    P.sx = (const float*)d_in[2];
    P.sw = (const float*)d_in[3];
    for (int p = 0; p < NR; ++p) {
        P.out[p] = (float*)d_out_all[p];
        P.ws[p] = (uint8_t*)d_ws_all[p];
    }
    P.rank = rank;
    fused_kernel<<<GRID, THREADS, LDS_BYTES, stream>>>(P);
    HIP_CHECK(hipGetLastError());
}
```
